```python
import numpy as np

kernel_name = "gpu_gemm_a2a_m4096_k4096_n4096_w8a8_dqrelu"


ABSMAX_RTOL = 0.001

COMPUTE_DTYPE = "int8"
EPILOGUE = "dequant_relu"
IN_QUANT = "int8"
OUT_QUANT = None
IN_DTYPES = {"x": "int8", "w_mat": "int8",
             "scale_x": "float32", "scale_w": "float32"}
WORLD_SIZE = 8

def setup_inputs():

    x = np.random.randint(-127, 128, size=(4096, 4096)).astype(np.int8)
    w_mat = np.random.randint(-127, 128, size=(4096, 4096)).astype(np.int8)
    scale_x = (0.02 * (1.0 + np.random.rand(1))).astype(np.float32)
    scale_w = (0.02 * (1.0 + np.random.rand(1))).astype(np.float32)
    return {"x": x, "w_mat": w_mat, "scale_x": scale_x, "scale_w": scale_w}

def reference(x, w_mat, scale_x, scale_w):
    acc = x.astype(np.float64) @ w_mat.astype(np.float64)
    y = (acc.astype(np.float32) * (scale_x * scale_w)).astype(np.float32)
    return np.maximum(y, 0.0).astype(np.float32)
```

```cpp
#include <hip/hip_runtime.h>
#include <rccl/rccl.h>
#include <cstdio>
#include <cstdint>

#define HIP_CHECK(expr) do { hipError_t _e = (expr); if (_e != hipSuccess) { \
    fprintf(stderr, "HIP error %s at %s:%d\n", hipGetErrorString(_e), __FILE__, __LINE__); \
    return; } } while (0)

typedef int v2i __attribute__((ext_vector_type(2)));
typedef int v4i __attribute__((ext_vector_type(4)));
typedef unsigned u32x4 __attribute__((ext_vector_type(4)));
typedef float f32x4 __attribute__((ext_vector_type(4)));

#define RBLK 5
#define MAPPING 0
#define INFL 3
#define NSEND 1
#define XINFL 3
#define ROT 1
#define DELAY 0x000149e
#define DELAY_L 14
#define PACE 968
#define PACE0 60
#define BURST 0
#define WARM 2
#define GAP 8
#define X_NT 0
#define LOCAL_AUX 1

constexpr int NR = 8;
constexpr int KD = 4096;
constexpr int ND = 4096;
constexpr int MLOC = 512;
constexpr int NCOL = 512;
constexpr int RB = RBLK;
constexpr int SA = (RB + 1) / 2;
constexpr int NWG = 256;
constexpr unsigned OUT_BYTES = 4096u * NCOL * 4u;
constexpr unsigned X_BYTES = MLOC * KD;
constexpr unsigned W_BYTES = (unsigned)KD * ND;
constexpr unsigned WS_CALL = 0;
constexpr unsigned WS_TMO = 0x1000;
constexpr unsigned WS_FLAG = 0x2000;
constexpr unsigned WS_XFLAG = 0x3000;
constexpr unsigned WS_XS = 0x10000;
constexpr unsigned XS_SLOT = RB * 16u * KD;
constexpr unsigned WS_END = WS_XS + NR * XS_SLOT;
constexpr unsigned long long DEADLINE_TICKS = 100000000ull;

constexpr unsigned ST_X = 16384u;
constexpr unsigned LDS_BYTES = 163840u;
#define SPB 1
__host__ __device__ constexpr unsigned stage_bytes(int mt) { return ST_X + mt * 2048u; }
__host__ __device__ constexpr int ring_steps(int mt) { return (int)(LDS_BYTES / (SPB * stage_bytes(mt))); }
constexpr int NKT = KD / 128;

#define SPLIT 0x1234566
__host__ __device__ constexpr int split_mt(int s) {
    if (SPLIT >= 0) return (SPLIT >> (4 * (6 - s))) & 15;
    return (32 - RB) / 7 + (s >= 7 - (32 - RB) % 7 ? 1 : 0);
}
__host__ __device__ constexpr int split_start(int s) { int a = RB; for (int i = 0; i < s; ++i) a += split_mt(i); return a; }
static_assert(split_start(7) == 32, "remote split must cover the shard");
static_assert(RB >= 2 && RB <= 6, "RB");

struct Params {
    const char* x;
    const char* w;
    const float* sx;
    const float* sw;
    char* out[NR];
    char* ws[NR];
    unsigned long long* dbg;
    int rank;
};

__device__ __forceinline__ void st_sys(void* p, unsigned v) {
    __hip_atomic_store((unsigned*)p, v, __ATOMIC_RELAXED, __HIP_MEMORY_SCOPE_SYSTEM);
}
__device__ __forceinline__ unsigned ld_agent(const void* p) {
    return __builtin_amdgcn_readfirstlane(__hip_atomic_load((const unsigned*)p, __ATOMIC_RELAXED, __HIP_MEMORY_SCOPE_AGENT));
}
__device__ __forceinline__ void st_agent(void* p, unsigned v) {
    __hip_atomic_store((unsigned*)p, v, __ATOMIC_RELAXED, __HIP_MEMORY_SCOPE_AGENT);
}

#define LDS_AS __attribute__((address_space(3)))
typedef LDS_AS char lds_char;
__device__ __forceinline__ v4i lds_ld(unsigned a) { return *(const LDS_AS v4i*)(lds_char*)(uintptr_t)a; }
__device__ __forceinline__ void lds_st(unsigned a, v4i v) { *(LDS_AS v4i*)(lds_char*)(uintptr_t)a = v; }
__device__ __forceinline__ v2i lds_tr(unsigned a) { return __builtin_amdgcn_ds_read_tr8_b64_v2i32((LDS_AS v2i*)(lds_char*)(uintptr_t)a); }

#define BARRIER() do { asm volatile("" ::: "memory"); __builtin_amdgcn_s_barrier(); asm volatile("" ::: "memory"); } while (0)
#define WAIT_VM(n) asm volatile("s_waitcnt vmcnt(%0)" :: "i"(n) : "memory")
#define WAIT_LGKM0() do { asm volatile("" ::: "memory"); __builtin_amdgcn_s_waitcnt(0xC07F); asm volatile("" ::: "memory"); } while (0)
#define DMA16(m0v, voff, rsrc, soff) \
    asm volatile("s_mov_b32 m0, %0\n\ts_nop 0\n\tbuffer_load_dwordx4 %1, %2, %3 offen lds" \
                 :: "s"(m0v), "v"(voff), "s"(rsrc), "s"(soff) : "memory")

#define DMA16X DMA16
#define STORE16_SYS(data, voff, rsrc, soff) \
    asm volatile("buffer_store_dwordx4 %0, %1, %2, %3 offen sc0 sc1\n\ts_nop 1" :: "v"(data), "v"(voff), "s"(rsrc), "s"(soff) : "memory")
#define STORE16(data, voff, rsrc, soff) \
    asm volatile("buffer_store_dwordx4 %0, %1, %2, %3 offen\n\ts_nop 1" :: "v"(data), "v"(voff), "s"(rsrc), "s"(soff) : "memory")

__device__ __forceinline__ void wait_due(unsigned tstart, int q, unsigned col) {
    const unsigned due = PACE0 * 128u + (unsigned)max(q - BURST, 0) * PACE + col * (PACE / 4);
    while (((unsigned)__builtin_amdgcn_s_memrealtime() - tstart) * 128u < due) __builtin_amdgcn_s_sleep(1);
}

struct Job {
    __amdgpu_buffer_rsrc_t rx, rw;
    unsigned xs;
    unsigned wcol;
    unsigned ldsb;
    int wid, lane;
    int rot;
};

constexpr int NSTEP = NKT / SPB;

template <int N, int PER>
__device__ __forceinline__ void drain_steps() {
    WAIT_VM(PER * (N - 1));
    BARRIER();
    if constexpr (N > 1) drain_steps<N - 1, PER>();
}

template <int MT, int NX>
__device__ __forceinline__ void load_tile(const Job& j) {
    constexpr unsigned STB = stage_bytes(MT);
    constexpr int NS0 = ring_steps(MT), PER = (4 + NX) * SPB;
    constexpr int NS = PER * (NS0 - 1) > 63 ? 63 / PER + 1 : NS0;
    static_assert(NS == NS0, "ring deeper than vmcnt can count");
    const unsigned lw = j.wid - 8, lane = j.lane;
    const unsigned l3 = lane >> 3, c8 = lane & 7;
    const unsigned sw = (c8 ^ ((l3 >> 1) | ((lw & 1) << 2))) * 16;
    const unsigned wv = l3 * ND + j.wcol + sw;
    const unsigned xv = l3 * KD + sw;
    const unsigned wm0 = j.ldsb + lw * 2048, xm0 = j.ldsb + ST_X + lw * 1024;
    const unsigned wso = lw * 16u * ND, xso = j.xs + lw * 8u * KD;
    auto step = [&](int n, unsigned sb) {
#pragma unroll
        for (int u = 0; u < SPB; ++u) {
            const int kt = (n * SPB + u + j.rot) & (NKT - 1);
            const unsigned a = wm0 + sb + u * STB, s = wso + kt * (128u * ND);
            const unsigned c = xm0 + sb + u * STB, t = xso + kt * 128u;
#pragma unroll
            for (int k = 0; k < 4; ++k) {
                const unsigned ak = a + (k & 1) * 1024 + (k >> 1) * 8192, sk = s + ((k & 1) * 8u + (k >> 1) * 64u) * ND;
                DMA16(ak, wv, j.rw, sk);
                if (k < NX) {
                    const unsigned ck = c + k * 4096, tk = t + k * 32u * KD;
                    DMA16X(ck, xv, j.rx, tk);
                }
            }
        }
    };
#pragma unroll
    for (int s = 0; s < NS; ++s) step(s, s * SPB * STB);
    WAIT_VM(PER * (NS - 1));
    BARRIER();
    unsigned sb = 0;
#pragma unroll 1
    for (int n = 1; n <= NSTEP - NS; ++n) {
        WAIT_VM(PER * (NS - 2));
        BARRIER();
        step(n - 1 + NS, sb);
        sb = sb + SPB * STB == NS * SPB * STB ? 0 : sb + SPB * STB;
    }
    drain_steps<NS - 1, PER>();
    BARRIER();
    BARRIER();
}

template <int MT>
struct Frags { v4i w[SPB][2], x[SPB][MT]; };

template <int MT>
__device__ __forceinline__ void mma_tile(const Job& j) {
    constexpr unsigned STB = stage_bytes(MT), RING = ring_steps(MT) * SPB * STB;
    const unsigned wid = j.wid, lane = j.lane;
    const unsigned cn = wid & 3, kh = wid >> 2;
    const unsigned lg = lane >> 4, m = lane & 15, q = m >> 1;
    unsigned wa[2];
#pragma unroll
    for (int t = 0; t < 2; ++t)
        wa[t] = j.ldsb + (8 * kh + 2 * lg) * 1024 + q * 128 + (((2 * cn + t) ^ ((q >> 1) | ((lg & 1) << 2))) * 16) + (m & 1) * 8;
    const unsigned xa = j.ldsb + ST_X + m * 128 + (((4 * kh + lg) ^ (m >> 1)) * 16);
    v4i acc[MT][2];
#pragma unroll
    for (int i = 0; i < MT; ++i) acc[i][0] = acc[i][1] = v4i{0, 0, 0, 0};
    auto rd = [&](Frags<MT>& f, unsigned sb) {
#pragma unroll
        for (int u = 0; u < SPB; ++u) {
#pragma unroll
            for (int t = 0; t < 2; ++t) {
                const v2i lo = lds_tr(wa[t] + sb + u * STB), hi = lds_tr(wa[t] + sb + u * STB + 1024);
                f.w[u][t] = v4i{lo.x, lo.y, hi.x, hi.y};
            }
#pragma unroll
            for (int i = 0; i < MT; ++i) f.x[u][i] = lds_ld(xa + sb + u * STB + i * 2048);
        }
    };
    auto mm = [&](const Frags<MT>& f) {
#pragma unroll
        for (int u = 0; u < SPB; ++u) {
#pragma unroll
            for (int i = 0; i < MT; ++i)
#pragma unroll
                for (int t = 0; t < 2; ++t)
                    acc[i][t] = __builtin_amdgcn_mfma_i32_16x16x64_i8(f.w[u][t], f.x[u][i], acc[i][t], 0, 0, 0);
        }
    };
    Frags<MT> fa, fb;
    unsigned sb = 0;
    auto next = [&]() { sb = sb + SPB * STB == RING ? 0 : sb + SPB * STB; };
    BARRIER();
    rd(fa, sb); next();
#pragma unroll 1
    for (int n = 1; n < NSTEP - 1; n += 2) {
        WAIT_LGKM0(); __builtin_amdgcn_sched_barrier(0);
        BARRIER();
        rd(fb, sb); next();
        __builtin_amdgcn_sched_barrier(0);
        mm(fa);
        __builtin_amdgcn_sched_barrier(0);
        WAIT_LGKM0(); __builtin_amdgcn_sched_barrier(0);
        BARRIER();
        rd(fa, sb); next();
        __builtin_amdgcn_sched_barrier(0);
        mm(fb);
        __builtin_amdgcn_sched_barrier(0);
    }
    WAIT_LGKM0();
    BARRIER();
    rd(fb, sb);
    __builtin_amdgcn_sched_barrier(0);
    mm(fa);
    __builtin_amdgcn_sched_barrier(0);
    WAIT_LGKM0();
    BARRIER();
    mm(fb);
    const unsigned pw = j.ldsb + kh * (MT * 8192) + m * 512;
#pragma unroll
    for (int i = 0; i < MT; ++i)
#pragma unroll
        for (int t = 0; t < 2; ++t)
            lds_st(pw + i * 8192 + ((((2 * cn + t) * 4 + lg) ^ m) * 16), acc[i][t]);
    WAIT_LGKM0();
    BARRIER();
}

template <int MT>
__device__ __forceinline__ void tile(const Job& j) {
    if (j.wid >= 8) {
        constexpr int HI = (2 * MT + 3) / 4, LO = (2 * MT) / 4;
        if (HI == LO || j.wid - 8 < (2 * MT) % 4) load_tile<MT, HI>(j);
        else load_tile<MT, LO>(j);
    } else {
        mma_tile<MT>(j);
    }
}

__global__ void __launch_bounds__(768, 3) gemm_a2a(Params p) {
    extern __shared__ __attribute__((aligned(16))) char lds[];
    const unsigned tstart = (unsigned)__builtin_amdgcn_s_memrealtime();
    const int lane = threadIdx.x & 63;
    const int wid = __builtin_amdgcn_readfirstlane(threadIdx.x >> 6);
    const int b = blockIdx.x;
    const int rank = p.rank;
    const int idx = b >> 3;
    const unsigned dr = (((b & 7) + 8 * ((idx >> 3) & 3)) >> 2) & 7;
    const unsigned col = b & 3;
    const unsigned s8 = idx & 7;
    const unsigned peer = (unsigned)(rank + dr) & (NR - 1);
    char* myws = p.ws[0];
    char* peerws = p.ws[dr];
    typedef const __attribute__((address_space(4))) float* cfp;
    const float scale = *(cfp)p.sx * *(cfp)p.sw;

    Job j;
    j.wid = wid; j.lane = lane;
    j.rot = ROT ? (dr != 0 ? 8 * (s8 & 3) : 4 * s8) : 0;
    j.ldsb = (unsigned)(uintptr_t)(lds_char*)lds;
    j.rx = __builtin_amdgcn_make_buffer_rsrc((void*)p.x, 0, X_BYTES, 0x00020000);
    j.rw = __builtin_amdgcn_make_buffer_rsrc((void*)p.w, 0, W_BYTES, 0x00020000);

    unsigned call = 0;
    if (dr != 0 && s8 == 7) {
        constexpr int NU = RB * 2;
        const unsigned u0 = (col * 8 * NU + (wid & 7)) * 1024u;
        auto rs = __builtin_amdgcn_make_buffer_rsrc(peerws + WS_XS + rank * XS_SLOT, 0, XS_SLOT, 0x00020000);
        if (wid < 8) {
            u32x4 pu[NU];
#pragma unroll
            for (int k = 0; k < NU; ++k)
                asm volatile("buffer_load_dwordx4 %0, %1, %2, %3 offen" : "=v"(pu[k]) : "v"(lane * 16), "s"(j.rx), "s"(u0 + k * 8192u) : "memory");
#pragma unroll
            for (int k = 0; k < NU; ++k) {
                wait_due(tstart, wid + 8 * k, col);
                asm volatile("s_waitcnt vmcnt(%1)" : "+v"(pu[k]) : "i"(k >= INFL ? INFL - 1 : NU - 1) : "memory");
                STORE16_SYS(pu[k], lane * 16, rs, u0 + k * 8192u);
            }
        }
        call = ld_agent(myws + WS_CALL + b * 4) + 1u;
        WAIT_VM(0);
        BARRIER();
        if (threadIdx.x == 0) st_sys(peerws + WS_XFLAG + (rank * 32 + col) * 4, call);
    } else {
        const unsigned dly = (dr != 0 ? ((DELAY >> (4 * (6 - s8))) & 15) : DELAY_L) * 50u;
        const unsigned third = (dr - 1) >> 1;
        if (WARM && dr != 0 && s8 >= 4 && third < 3) {
            unsigned d0, d1;
            const unsigned e = (third * 3 + (s8 - 4)) * 768u + threadIdx.x;
            asm volatile("buffer_load_dword %0, %1, %2, 0 offen\n\ts_waitcnt vmcnt(0)"
                         : "=v"(d0) : "v"(e < (split_start(4) - RB) * 512u ? RB * 16u * KD + e * 128u : 0xffffffffu), "s"(j.rx) : "memory");
            if (WARM > 1) {
                const unsigned st = s8 == 4 ? split_start(4) : s8 == 5 ? split_start(5) : split_start(6);
                const unsigned n = (s8 == 4 ? split_mt(4) : s8 == 5 ? split_mt(5) : split_mt(6)) * 512u;
                while ((unsigned)__builtin_amdgcn_s_memrealtime() - tstart + 100u < dly) __builtin_amdgcn_s_sleep(8);
                const unsigned l0 = third * 768u + threadIdx.x, l1 = l0 + 2304u;
                asm volatile("buffer_load_dword %0, %2, %4, 0 offen\n\tbuffer_load_dword %1, %3, %4, 0 offen\n\ts_waitcnt vmcnt(0)"
                             : "=&v"(d0), "=&v"(d1)
                             : "v"(l0 < n ? st * 16u * KD + l0 * 128u : 0xffffffffu), "v"(l1 < n ? st * 16u * KD + l1 * 128u : 0xffffffffu), "s"(j.rx) : "memory");
            }
        }
        while ((unsigned)__builtin_amdgcn_s_memrealtime() - tstart < dly) __builtin_amdgcn_s_sleep(8);
    }

    const int njob = (dr != 0 && s8 == 0 && RB > SA) ? 2 : 1;
#pragma unroll 1
    for (int jb = 0; jb < njob; ++jb) {
        const bool staged = dr != 0 && (s8 == 7 || jb == 1);
        const bool remote = dr != 0 && !staged;
        int mt, blk0;
        if (remote) {
            mt = split_mt(0); blk0 = split_start(0);
#pragma unroll
            for (int s = 1; s < 7; ++s) if (s8 == (unsigned)s) { mt = split_mt(s); blk0 = split_start(s); }
        } else if (staged) { mt = jb ? RB - SA : SA; blk0 = jb ? SA : 0; }
        else { mt = 4; blk0 = 4 * s8; }
        asm volatile("" : "+s"(j.ldsb), "+s"(j.wid), "+v"(j.lane));
        j.xs = blk0 * 16u * KD;
        j.wcol = (remote ? peer : (unsigned)rank) * NCOL + col * 128;
        const unsigned orow = (staged ? peer : (unsigned)rank) * MLOC + blk0 * 16u;
        if (staged) {
            if (wid == 0) {
                const unsigned long long t0 = __builtin_amdgcn_s_memrealtime();
                const char* f = myws + WS_XFLAG + (peer * 32 + (lane & 3)) * 4;
                while (true) {
                    const unsigned v = __hip_atomic_load((const unsigned*)f, __ATOMIC_RELAXED, __HIP_MEMORY_SCOPE_SYSTEM);
                    if (__all(v >= call)) break;
                    if (ld_agent(myws + WS_TMO) == call || __builtin_amdgcn_s_memrealtime() - t0 > DEADLINE_TICKS) {
                        st_agent(myws + WS_TMO, call);
                        break;
                    }
                    __builtin_amdgcn_s_sleep(2);
                }
                __builtin_amdgcn_fence(__ATOMIC_ACQUIRE, "");
                WAIT_VM(0);
            }
            BARRIER();
            j.rx = __builtin_amdgcn_make_buffer_rsrc(myws + WS_XS + peer * XS_SLOT, 0, XS_SLOT, 0x00020000);
        }
        switch (mt) {
            case 1: tile<1>(j); break;
            case 2: tile<2>(j); break;
            case 3: tile<3>(j); break;
            case 4: tile<4>(j); break;
            case 5: tile<5>(j); break;
            default: tile<6>(j); break;
        }
        if (wid < 8) {
            auto ro = __builtin_amdgcn_make_buffer_rsrc(remote ? p.out[dr] : p.out[0], 0, OUT_BYTES, 0x00020000);
            const unsigned rl = lane >> 5, ch = lane & 31;
            const unsigned ovoff = rl * (NCOL * 4) + col * 512 + ch * 16;
            const unsigned poff = mt * 8192u;
            const bool bad = staged && ld_agent(myws + WS_TMO) == call;
#pragma unroll 1
            for (int k = wid; k < mt * 8; k += 8) {
                const unsigned r = 2 * k + rl;
                const unsigned a = j.ldsb + r * 512 + ((ch ^ (r & 15)) * 16);
                const v4i v = lds_ld(a) + lds_ld(a + poff);
                f32x4 f;
#pragma unroll
                for (int e = 0; e < 4; ++e) f[e] = fmaxf((float)v[e] * scale, 0.0f);
                if (bad) f = f32x4{__builtin_nanf(""), __builtin_nanf(""), __builtin_nanf(""), __builtin_nanf("")};
                const unsigned so = (orow + 2 * k) * (NCOL * 4);
                if (remote) {
                    asm volatile("" : "+v"(f));
                    wait_due(tstart, 8 * (RB + blk0) + k + GAP, col);
                    WAIT_VM(INFL - 1);
                    STORE16_SYS(f, ovoff, ro, so);
                } else if (LOCAL_AUX) {
                    STORE16_SYS(f, ovoff, ro, so);
                } else {
                    STORE16(f, ovoff, ro, so);
                }
            }
        }
        if (!staged) call = ld_agent(myws + WS_CALL + b * 4) + 1u;
        if (remote) WAIT_VM(0);
        BARRIER();
    }
    if (dr != 0 && threadIdx.x == 0) st_sys(peerws + WS_FLAG + (rank * 32 + col * 8 + s8) * 4, call);
    if (wid != 0) return;
    if (lane == 0) st_agent(myws + WS_CALL + b * 4, call);
    if (b == 0) {
        const unsigned long long t0 = __builtin_amdgcn_s_memrealtime();
        auto rf = __builtin_amdgcn_make_buffer_rsrc(myws + WS_FLAG, 0, NR * 128, 0x00020000);
        for (unsigned it = 1;; ++it) {
            const u32x4 v = __builtin_amdgcn_raw_buffer_load_b128(rf, lane * 16, 0, 17);
            const bool ok = (lane >> 3) == rank || (v.x >= call && v.y >= call && v.z >= call && v.w >= call);
            if (__all(ok)) break;
            if ((it & 255) == 0 &&
                (ld_agent(myws + WS_TMO) == call || __builtin_amdgcn_s_memrealtime() - t0 > DEADLINE_TICKS)) {
                st_agent(myws + WS_TMO, call);
                WAIT_VM(0);
                if (lane == 0) *(float*)(p.out[0] + (size_t)(rank * MLOC) * (NCOL * 4)) = __builtin_nanf("");
                break;
            }
        }
    }
}

extern "C" void kernel_launch(int rank, int nranks, ncclComm_t comm,
                              hipStream_t stream,
                              void* const* d_in, const int* in_sizes, int n_in,
                              void* d_out, int out_size,
                              const int* in_dtypes, int out_dtype,
                              void* const* d_out_all,
                              void* const* d_ws_all, size_t ws_bytes) {
    if (nranks != NR || (unsigned)rank >= (unsigned)NR || n_in < 4 || ws_bytes < WS_END ||
        in_sizes[0] != (int)X_BYTES || in_sizes[1] != (int)W_BYTES || out_size != 4096 * NCOL) {
        fprintf(stderr, "kernel_launch: unexpected problem shape\n");
        return;
    }
    HIP_CHECK(hipFuncSetAttribute((const void*)gemm_a2a, hipFuncAttributeMaxDynamicSharedMemorySize, LDS_BYTES));
    Params p;
    p.x = (const char*)d_in[0];
    p.w = (const char*)d_in[1];
    p.sx = (const float*)d_in[2];
    p.sw = (const float*)d_in[3];
    for (int i = 0; i < NR; ++i) { p.out[i] = (char*)d_out_all[(rank + i) % NR]; p.ws[i] = (char*)d_ws_all[(rank + i) % NR]; }
    p.dbg = nullptr;
    p.rank = rank;
    gemm_a2a<<<NWG, 768, LDS_BYTES, stream>>>(p);
    HIP_CHECK(hipGetLastError());
}
```

```cpp
#include <hip/hip_runtime.h>
#include <rccl/rccl.h>
#include <cstdio>
#include <cstdint>

#define HIP_CHECK(expr) do { hipError_t _e = (expr); if (_e != hipSuccess) { \
    fprintf(stderr, "HIP error %s at %s:%d\n", hipGetErrorString(_e), __FILE__, __LINE__); \
    return; } } while (0)

typedef int v2i __attribute__((ext_vector_type(2)));
typedef int v4i __attribute__((ext_vector_type(4)));
typedef unsigned u32x4 __attribute__((ext_vector_type(4)));
typedef float f32x4 __attribute__((ext_vector_type(4)));
typedef unsigned long long u64;

#define RBLK 7
#define SPLIT 0x1234555
#define DELAY 0x00015af
#define DELAY_L 14
#define INFL 3
#define PACE 912
#define LEAD 70
#define STAG_MIN 4
#define LEAD_MT 15
#define MAP2 1
#define EARLY_ACQ 0
#define PUSH0 70
#define RES_AT 27
#define WARM 15
#define WARM_TILES 5

constexpr int NR = 8;
constexpr int KD = 4096;
constexpr int ND = 4096;
constexpr int MLOC = 512;
constexpr int NCOL = 512;
constexpr int RB = RBLK;
constexpr int SA = (RB + 1) / 2;
constexpr int NWG = 256;
constexpr unsigned OUT_BYTES = 4096u * NCOL * 4u;
constexpr unsigned X_BYTES = MLOC * KD;
constexpr unsigned W_BYTES = (unsigned)KD * ND;
constexpr unsigned WS_CALL = 0;
constexpr unsigned WS_TMO = 0x1000;
constexpr unsigned WS_FLAG = 0x2000;
constexpr unsigned WS_XFLAG = 0x3000;
constexpr unsigned WS_TT = 0x4000;
constexpr unsigned WS_XS = 0x10000;
constexpr unsigned XS_SLOT = RB * 16u * KD;
constexpr unsigned WS_END = WS_XS + NR * XS_SLOT;
constexpr u64 DEADLINE_TICKS = 100000000ull;

constexpr unsigned ST_X = 16384u;
constexpr unsigned STB = 28672u;
constexpr unsigned RING = 4u * STB;
constexpr unsigned LDS_T0 = RING;
constexpr unsigned LDS_DUMMY = RING + 256u;
constexpr unsigned LDS_BYTES = RING + 512u;
constexpr int NKT = KD / 128;

__host__ __device__ constexpr int split_mt(int s) { return (SPLIT >> (4 * (6 - s))) & 15; }
__host__ __device__ constexpr int split_start(int s) { int a = RB; for (int i = 0; i < s; ++i) a += split_mt(i); return a; }
__host__ __device__ constexpr u64 pack_starts() { u64 v = 0; for (int s = 0; s < 7; ++s) v |= (u64)split_start(s) << (8 * s); return v; }
static_assert(split_start(7) == 32, "remote split must cover the shard");
static_assert(RB >= 2 && RB <= 12, "RB");

struct Params {
    const char* x;
    const char* w;
    int rank;
    int pad;
    const float* sx;
    const float* sw;
    char* out[NR];
    char* ws[NR];
    u64* dbg;
};

__device__ __forceinline__ void st_sys(void* p, unsigned v) {
    __hip_atomic_store((unsigned*)p, v, __ATOMIC_RELAXED, __HIP_MEMORY_SCOPE_SYSTEM);
}
__device__ __forceinline__ unsigned ld_agent(const void* p) {
    return __builtin_amdgcn_readfirstlane(__hip_atomic_load((const unsigned*)p, __ATOMIC_RELAXED, __HIP_MEMORY_SCOPE_AGENT));
}
__device__ __forceinline__ void st_agent(void* p, unsigned v) {
    __hip_atomic_store((unsigned*)p, v, __ATOMIC_RELAXED, __HIP_MEMORY_SCOPE_AGENT);
}
__device__ __forceinline__ u64 now() { return __builtin_amdgcn_s_memrealtime(); }

#define LDS_AS __attribute__((address_space(3)))
typedef LDS_AS char lds_char;
__device__ __forceinline__ v4i lds_ld(unsigned a) { return *(const LDS_AS v4i*)(lds_char*)(uintptr_t)a; }
__device__ __forceinline__ void lds_st(unsigned a, v4i v) { *(LDS_AS v4i*)(lds_char*)(uintptr_t)a = v; }
__device__ __forceinline__ v2i lds_tr(unsigned a) { return __builtin_amdgcn_ds_read_tr8_b64_v2i32((LDS_AS v2i*)(lds_char*)(uintptr_t)a); }

#define BARRIER() do { asm volatile("" ::: "memory"); __builtin_amdgcn_s_barrier(); asm volatile("" ::: "memory"); } while (0)
#define WAIT_VM(n) asm volatile("s_waitcnt vmcnt(%0)" :: "i"(n) : "memory")
#define WAIT_LGKM0() do { asm volatile("" ::: "memory"); __builtin_amdgcn_s_waitcnt(0xC07F); asm volatile("" ::: "memory"); } while (0)
#define DMA16(m0v, voff, rsrc, soff) \
    asm volatile("s_mov_b32 m0, %0\n\ts_nop 0\n\tbuffer_load_dwordx4 %1, %2, %3 offen lds" \
                 :: "s"(m0v), "v"(voff), "s"(rsrc), "s"(soff) : "memory")
#define STORE16_SYS(data, voff, rsrc, soff) \
    asm volatile("buffer_store_dwordx4 %0, %1, %2, %3 offen sc0 sc1\n\ts_nop 1" :: "v"(data), "v"(voff), "s"(rsrc), "s"(soff) : "memory")
#define TOUCH(voff, rsrc) \
    asm volatile("s_mov_b32 m0, %0\n\ts_nop 0\n\tbuffer_load_dword %1, %2, 0 offen lds" :: "s"(ldsb + LDS_DUMMY), "v"(voff), "s"(rsrc) : "memory")

__device__ __forceinline__ void reserve(u64& slot, char* tt, u64 tmin, unsigned n) {
    const u64 a = tmin * 128u, d = (u64)n * PACE;
    asm volatile("global_atomic_umax_x2 %1, %2, %4\n\t"
                 "global_atomic_add_x2 %0, %1, %3, %4 sc0\n\ts_nop 1"
                 : "+v"(slot) : "v"(0u), "v"(a), "v"(d), "s"(tt) : "memory");
}
__device__ __forceinline__ void wait_until(unsigned due) {
    for (unsigned d = due - (unsigned)now(); d - 1u < 10000u; d = due - (unsigned)now()) __builtin_amdgcn_s_sleep(1);
}

#define FINE(k)

struct Ld {
    __amdgpu_buffer_rsrc_t rx, rw;
    unsigned wv[4], xv[3];
    unsigned a, c;
    unsigned kt;
};
template <int NX>
__device__ __forceinline__ void ld_stage(const Ld& l, unsigned sb, unsigned kt) {
    const unsigned a = l.a + sb, c = l.c + sb, s = kt * (128u * ND), t = kt * 128u;
    if (NX == 1)
        asm volatile("s_mov_b32 m0, %0\n\ts_nop 0\n\tbuffer_load_dwordx4 %1, %2, %3 offen lds"
                     :: "s"(c), "v"(l.xv[0]), "s"(l.rx), "s"(t) : "memory");
    if (NX == 2)
        asm volatile("s_mov_b32 m0, %0\n\ts_nop 0\n\tbuffer_load_dwordx4 %1, %3, %4 offen lds\n\t"
                     "buffer_load_dwordx4 %2, %3, %4 offen offset:1024 lds"
                     :: "s"(c), "v"(l.xv[0]), "v"(l.xv[1]), "s"(l.rx), "s"(t) : "memory");
    if (NX == 3)
        asm volatile("s_mov_b32 m0, %0\n\ts_nop 0\n\tbuffer_load_dwordx4 %1, %4, %5 offen lds\n\t"
                     "buffer_load_dwordx4 %2, %4, %5 offen offset:1024 lds\n\t"
                     "buffer_load_dwordx4 %3, %4, %5 offen offset:2048 lds"
                     :: "s"(c), "v"(l.xv[0]), "v"(l.xv[1]), "v"(l.xv[2]), "s"(l.rx), "s"(t) : "memory");
    asm volatile("s_mov_b32 m0, %0\n\ts_nop 0\n\tbuffer_load_dwordx4 %1, %5, %6 offen lds\n\t"
                 "buffer_load_dwordx4 %2, %5, %6 offen offset:1024 lds\n\t"
                 "buffer_load_dwordx4 %3, %5, %6 offen offset:2048 lds\n\t"
                 "buffer_load_dwordx4 %4, %5, %6 offen offset:3072 lds"
                 :: "s"(a), "v"(l.wv[0]), "v"(l.wv[1]), "v"(l.wv[2]), "v"(l.wv[3]), "s"(l.rw), "s"(s) : "memory");
}
template <int NX>
__device__ __forceinline__ void load_tile(const Ld& l) {
    constexpr int PER = 4 + NX;
    unsigned kt = l.kt;
    ld_stage<NX>(l, 0, kt); ld_stage<NX>(l, STB, kt + 1); ld_stage<NX>(l, 2 * STB, kt + 2);
    kt += 3;
    unsigned sb = 3 * STB;
#pragma unroll 1
    for (int n = 0; n < NKT - 3; ++n) {
        WAIT_VM(2 * PER); BARRIER();
        ld_stage<NX>(l, sb, kt);
        kt = (kt + 1) & (NKT - 1);
        sb = sb == 3 * STB ? 0 : sb + STB;
    }
    WAIT_VM(2 * PER); BARRIER();
    WAIT_VM(PER); BARRIER();
    WAIT_VM(0); BARRIER();
}

template <int MT>
struct Frags { v4i w[2], x[MT]; };

template <int MT>
__device__ __forceinline__ void mma_tile(unsigned ldsb, unsigned wid, unsigned lane, char* tt) {
    constexpr bool STAG = MT >= STAG_MIN;
    const unsigned cn = wid & 3, kh = wid >> 2;
    const unsigned lg = lane >> 4, m = lane & 15, q = m >> 1;
    unsigned wa[2];
#pragma unroll
    for (int t = 0; t < 2; ++t)
        wa[t] = ldsb + (8 * kh + 2 * lg) * 1024 + q * 128 + (((2 * cn + t) ^ ((q >> 1) | ((lg & 1) << 2))) * 16) + (m & 1) * 8;
    const unsigned xa = ldsb + ST_X + m * 128 + (((4 * kh + lg) ^ (m >> 1)) * 16);
    v4i acc[MT][2];
#pragma unroll
    for (int i = 0; i < MT; ++i) acc[i][0] = acc[i][1] = v4i{0, 0, 0, 0};
    auto rd = [&](Frags<MT>& f, unsigned sb) {
#pragma unroll
        for (int t = 0; t < 2; ++t) {
            const v2i lo = lds_tr(wa[t] + sb), hi = lds_tr(wa[t] + sb + 1024);
            f.w[t] = v4i{lo.x, lo.y, hi.x, hi.y};
        }
#pragma unroll
        for (int i = 0; i < MT; ++i) f.x[i] = lds_ld(xa + sb + i * 2048);
    };
    auto mm = [&](const Frags<MT>& f) {
#pragma unroll
        for (int i = 0; i < MT; ++i)
#pragma unroll
            for (int t = 0; t < 2; ++t)
                acc[i][t] = __builtin_amdgcn_mfma_i32_16x16x64_i8(f.w[t], f.x[i], acc[i][t], 0, 0, 0);
    };
    Frags<MT> fa, fb;
    u64 slot = 0;
    BARRIER();
    rd(fa, 0);
    if (STAG && kh) {
        unsigned sb = STB;
#pragma unroll 1
        for (int n = 1; n < NKT; ++n) {
            WAIT_LGKM0(); __builtin_amdgcn_sched_barrier(0);
            BARRIER();
            mm(fa);
            __builtin_amdgcn_sched_barrier(0);
            rd(fa, sb);
            __builtin_amdgcn_sched_barrier(0);
            sb = sb == 3 * STB ? 0 : sb + STB;
        }
        if (MT * 16384u > 3 * STB) {
            WAIT_LGKM0();
            BARRIER();
        }
        mm(fa);
    } else {
        unsigned sb = STB;
#pragma unroll 1
        for (int n = 1; n < NKT - 1; n += 2) {
            WAIT_LGKM0(); __builtin_amdgcn_sched_barrier(0);
            BARRIER();
            rd(fb, sb);
            __builtin_amdgcn_sched_barrier(0);
            mm(fa);
            __builtin_amdgcn_sched_barrier(0);
            sb = 4 * STB - sb;
            if (n == RES_AT && tt && lane == 0) reserve(slot, tt, now() + LEAD + (MT - 1) * LEAD_MT, MT * 8);
            WAIT_LGKM0(); __builtin_amdgcn_sched_barrier(0);
            BARRIER();
            rd(fa, sb - STB);
            __builtin_amdgcn_sched_barrier(0);
            mm(fb);
            __builtin_amdgcn_sched_barrier(0);
        }
        WAIT_LGKM0();
        BARRIER();
        rd(fb, 3 * STB);
        __builtin_amdgcn_sched_barrier(0);
        mm(fa);
        __builtin_amdgcn_sched_barrier(0);
        if (MT * 16384u > 3 * STB) {
            WAIT_LGKM0();
            BARRIER();
        }
        mm(fb);
    }
    const unsigned pw = ldsb + kh * (MT * 8192) + m * 512;
#pragma unroll
    for (int i = 0; i < MT; ++i)
#pragma unroll
        for (int t = 0; t < 2; ++t)
            lds_st(pw + i * 8192 + ((((2 * cn + t) * 4 + lg) ^ m) * 16), acc[i][t]);
    if (tt) {
        asm volatile("s_waitcnt vmcnt(0)" : "+v"(slot) :: "memory");
        if (lane == 0) *(volatile LDS_AS u64*)(lds_char*)(uintptr_t)(ldsb + LDS_T0) = slot;
    }
    WAIT_LGKM0();
    BARRIER();
}

__global__ void __launch_bounds__(768, 3) gemm_a2a(Params p) {
    extern __shared__ __attribute__((aligned(16))) char lds[];
    const u64 tstart = now();
    unsigned lane = threadIdx.x & 63;
    unsigned wid = __builtin_amdgcn_readfirstlane(threadIdx.x >> 6);
    const unsigned b = blockIdx.x;
    const unsigned rank = p.rank;
    const unsigned dr = ((b >> 2) & 1) | ((b >> 5) & 6);
    const unsigned lcol = b & 3;
    const unsigned col = (b - MAP2 * (dr >> 1)) & 3;
    const unsigned s8 = (b >> 3) & 7;
    const unsigned peer = (rank + dr) & (NR - 1);
    unsigned ldsb = (unsigned)(uintptr_t)(lds_char*)lds;
    auto rx = __builtin_amdgcn_make_buffer_rsrc((void*)p.x, 0, X_BYTES, 0x00020000);
    auto rw = __builtin_amdgcn_make_buffer_rsrc((void*)p.w, 0, W_BYTES, 0x00020000);
    const bool pusher = dr != 0 && s8 == 7;
    const unsigned dly = (dr != 0 ? (((unsigned)DELAY << 4) >> (4 * (7 - s8))) & 15 : DELAY_L) * 50u;

    const int njob = (dr != 0 && s8 == 0) ? 2 : 1;

    if (wid >= 8) {
        const unsigned lw = wid - 8;
#pragma unroll 1
        for (int jb = 0; jb < njob; ++jb) {
            const bool staged = dr != 0 && (s8 == 7 || jb == 1);
            const bool remote = dr != 0 && !staged;
            unsigned mt, blk0;
            if (remote) { mt = ((unsigned)SPLIT >> (4 * (6 - s8))) & 15; blk0 = (unsigned)(pack_starts() >> (8 * s8)) & 255; }
            else if (staged) { mt = jb ? RB - SA : SA; blk0 = jb ? SA : 0; }
            else { mt = 4; blk0 = 4 * s8; }
            const unsigned l3 = lane >> 3, c8 = lane & 7;
            const unsigned nx = (2 * mt + 3 - lw) >> 2;
            const unsigned x0 = lw * (mt >> 1) + min(lw, (2 * mt) & 3);
            Ld l;
            l.rx = rx; l.rw = rw;
            if (staged) l.rx = __builtin_amdgcn_make_buffer_rsrc(p.ws[0] + WS_XS + peer * XS_SLOT, 0, XS_SLOT, 0x00020000);
            const unsigned v0 = l3 * KD + ((c8 ^ (l3 >> 1)) * 16);
            const unsigned wb = v0 + lw * 32u * ND + (remote ? peer * NCOL + col * 128 : rank * NCOL + lcol * 128);
#pragma unroll
            for (int k = 0; k < 4; ++k) l.wv[k] = (wb ^ ((k >> 1) * 64)) + k * (8u * ND - 1024u);
#pragma unroll
            for (int k = 0; k < 3; ++k) l.xv[k] = (v0 ^ (((x0 + k) & 1) * 64)) + (blk0 * 16u + x0 * 8u) * KD + k * (8u * KD - 1024u);
            l.a = ldsb + lw * 4096; l.c = ldsb + ST_X + x0 * 1024;
            l.kt = dr != 0 ? 8 * (s8 & 3) : 4 * s8;
            if (EARLY_ACQ && s8 == 7 && dr != 0 && lw == 0) {
                __builtin_amdgcn_fence(__ATOMIC_ACQUIRE, "");
                WAIT_VM(0);
            }
            if ((WARM & 32) && s8 == 7 && dr != 0) {
                while ((unsigned)(now() - tstart) < 500u) __builtin_amdgcn_s_sleep(8);
                for (unsigned e = 0; e < 4; ++e)
                    TOUCH(((dr >> 1) * 1024u + e * 256u + lw * 64u + lane) * ND + rank * NCOL + lcol * 128, rw);
            }
            if (staged || dly) BARRIER();
            FINE(1);
            if (nx == 1) load_tile<1>(l);
            else if (nx == 0) load_tile<0>(l);
            else if (nx == 2) load_tile<2>(l);
            else load_tile<3>(l);
            FINE(6);
            if (mt * 16384u > 3 * STB) BARRIER();
            BARRIER();
            FINE(7);
            if (jb + 1 < njob) BARRIER();
        }
        return;
    }

    if (EARLY_ACQ && wid == 7 && dr != 0 && s8 == 0) __builtin_amdgcn_fence(__ATOMIC_ACQUIRE, "");
    unsigned call = 0;
    if (pusher) {
        char* myws = p.ws[0];
        char* peerws = p.ws[dr];
        constexpr int NU = RB * 2;
        const unsigned u0 = (col * 8 * NU + wid) * 1024u;
        constexpr int D = 4;
        u32x4 pu[D];
#define LOAD_UNIT(k) asm volatile("buffer_load_dwordx4 %0, %1, %2, %3 offen" : "=v"(pu[(k) % D]) : "v"(lane * 16), "s"(rx), "s"(u0 + (k) * 8192u) : "memory")
#pragma unroll
        for (int k = 0; k < D; ++k) LOAD_UNIT(k);
        auto rs = __builtin_amdgcn_make_buffer_rsrc(peerws + WS_XS + rank * XS_SLOT, 0, XS_SLOT, 0x00020000);
        const u64 t0 = tstart + PUSH0;
        if (wid == 7 && lane == 0)
            __hip_atomic_fetch_max((u64*)(myws + WS_TT + (dr * 4 + col) * 128), t0 * 128u + 8u * NU * PACE, __ATOMIC_RELAXED, __HIP_MEMORY_SCOPE_AGENT);
#pragma unroll
        for (int k = 0; k < NU; ++k) {
            wait_until((unsigned)t0 + (((8 * k + wid) * PACE) >> 7));
            constexpr int dummy = 0; (void)dummy;
            const int yl = (k + D - 1 < NU ? D - 1 : NU - 1 - k), ys = (k < D - 1 ? k : D - 1);
            asm volatile("s_waitcnt vmcnt(%1)" : "+v"(pu[k % D]) : "i"(yl + ys) : "memory");
            STORE16_SYS(pu[k % D], lane * 16, rs, u0 + k * 8192u);
            if (k + D < NU) LOAD_UNIT(k + D);
        }
#undef LOAD_UNIT
        call = ld_agent(myws + WS_CALL + b * 4) + 1u;
        WAIT_VM(0);
        if (lane == 0) st_sys(peerws + WS_XFLAG + (rank * 32 + col * 8 + wid) * 4, call);
    } else if (dly) {
        const unsigned third = (dr - 1) >> 1;
        if ((WARM & 8) && (s8 == 5 || s8 == 6)) {
            const unsigned cb = (3u - (unsigned)((uintptr_t)p.w >> 7) - 4u * peer) & 7u;
            const unsigned ib = (b - cb) & 3, db = (dr & 1) | (ib << 1);
            if (cb < 4 && db != 0) {
                const unsigned t = threadIdx.x, h = (dr >> 1) * 2 + (s8 - 5);
                TOUCH(((t & 3) * 1024u + (t >> 2) * 8u + h) * ND + ((rank + db) & (NR - 1)) * NCOL + cb * 128, rw);
            }
        }
        if (dr != 0 && s8 >= 5) {
            if ((WARM & 1) && third < 3) {
                constexpr unsigned NE = (split_start(WARM_TILES) - RB) * 512u;
                for (unsigned e = (third * 2 + (s8 - 5)) * 512u + threadIdx.x; e - threadIdx.x < NE; e += 3072u)
                    TOUCH(e < NE ? RB * 16u * KD + e * 128u : ~0u, rx);
            }
            if (WARM & 2) {
                const unsigned o = ((s8 - 5) * 2048u + 512u + threadIdx.x) * ND + peer * NCOL + col * 128;
                TOUCH(o, rw);
                TOUCH(o + 1024u * ND, rw);
            }
            if ((WARM & 4) && third < 3) {
                const unsigned st = (unsigned)(pack_starts() >> (8 * s8)) & 255, n = (((unsigned)SPLIT >> (4 * (6 - s8))) & 15) * 512u;
                while ((unsigned)(now() - tstart) + 100u < dly) __builtin_amdgcn_s_sleep(8);
                const unsigned l0 = third * 512u + threadIdx.x, l1 = l0 + 1536u;
                TOUCH(l0 < n ? st * 16u * KD + l0 * 128u : ~0u, rx);
                TOUCH(l1 < n ? st * 16u * KD + l1 * 128u : ~0u, rx);
            }
        }
        while ((unsigned)(now() - tstart) < dly) __builtin_amdgcn_s_sleep(8);
    }

#pragma unroll 1
    for (int jb = 0; jb < njob; ++jb) {
        const bool staged = dr != 0 && (s8 == 7 || jb == 1);
        const bool remote = dr != 0 && !staged;
        unsigned mt, blk0;
        if (remote) { mt = ((unsigned)SPLIT >> (4 * (6 - s8))) & 15; blk0 = (unsigned)(pack_starts() >> (8 * s8)) & 255; }
        else if (staged) { mt = jb ? RB - SA : SA; blk0 = jb ? SA : 0; }
        else { mt = 4; blk0 = 4 * s8; }
        asm volatile("" : "+s"(ldsb), "+s"(wid), "+v"(lane));
        if (staged && wid == 0) {
            char* myws = p.ws[0];
            const u64 t0 = now();
            const char* f = myws + WS_XFLAG + (peer * 32 + (lane & 31)) * 4;
            for (unsigned it = 1;; ++it) {
                const unsigned v = __hip_atomic_load((const unsigned*)f, __ATOMIC_RELAXED, __HIP_MEMORY_SCOPE_SYSTEM);
                if (__all(v >= call)) break;
                if ((it & 63) == 0 && (ld_agent(myws + WS_TMO) == call || now() - t0 > DEADLINE_TICKS)) {
                    st_agent(myws + WS_TMO, call);
                    break;
                }
            }
            if (!EARLY_ACQ) __builtin_amdgcn_fence(__ATOMIC_ACQUIRE, "");
        }
        if (staged) WAIT_VM(0);
        if (staged || dly) BARRIER();
        if (staged && (WARM & 16)) {
            auto rs = __builtin_amdgcn_make_buffer_rsrc(p.ws[0] + WS_XS + peer * XS_SLOT, 0, XS_SLOT, 0x00020000);
            for (unsigned e = 0; e < mt; ++e) TOUCH((blk0 + e) * 16u * KD + (wid * 64 + lane) * 128u, rs);
        }
        char* tt = (remote && wid == 0) ? p.ws[0] + WS_TT + (dr * 4 + col) * 128 : nullptr;
        switch (mt) {
            case 1: mma_tile<1>(ldsb, wid, lane, tt); break;
            case 2: mma_tile<2>(ldsb, wid, lane, tt); break;
            case 3: mma_tile<3>(ldsb, wid, lane, tt); break;
            case 4: mma_tile<4>(ldsb, wid, lane, tt); break;
            case 5: mma_tile<5>(ldsb, wid, lane, tt); break;
            default: mma_tile<6>(ldsb, wid, lane, tt); break;
        }
        typedef const __attribute__((address_space(4))) float* cfp;
        const float scale = *(cfp)p.sx * *(cfp)p.sw;
        auto ro = __builtin_amdgcn_make_buffer_rsrc(remote ? p.out[dr] : p.out[0], 0, OUT_BYTES, 0x00020000);
        const unsigned orow = (staged ? peer : rank) * MLOC + blk0 * 16u;
        const unsigned rl = lane >> 5, ch = lane & 31;
        const unsigned ovoff = rl * (NCOL * 4) + (remote ? col : lcol) * 512 + ch * 16;
        const bool bad = staged && ld_agent(p.ws[0] + WS_TMO) == call;
        unsigned t0 = 0;
        if (remote) t0 = __builtin_amdgcn_readfirstlane((unsigned)(*(volatile LDS_AS u64*)(lds_char*)(uintptr_t)(ldsb + LDS_T0) >> 7));
#pragma unroll 1
        for (unsigned k = wid; k < mt * 8; k += 8) {
            const unsigned r = 2 * k + rl;
            const unsigned a = ldsb + r * 512 + ((ch ^ (r & 15)) * 16);
            const v4i v = lds_ld(a) + lds_ld(a + mt * 8192u);
            f32x4 f;
#pragma unroll
            for (int e = 0; e < 4; ++e) f[e] = fmaxf((float)v[e] * scale, 0.0f);
            if (bad) f = f32x4{__builtin_nanf(""), __builtin_nanf(""), __builtin_nanf(""), __builtin_nanf("")};
            if (remote) {
                asm volatile("" : "+v"(f));
                wait_until(t0 + ((k * PACE) >> 7));
                WAIT_VM(INFL - 1);
            }
            STORE16_SYS(f, ovoff, ro, (orow + 2 * k) * (NCOL * 4));
        }
        if (!staged && (wid == 0 || njob == 2)) call = ld_agent(p.ws[0] + WS_CALL + b * 4) + 1u;
        if (remote) WAIT_VM(0);
        BARRIER();
    }
    if (wid != 0) return;
    char* myws = p.ws[0];
    if (lane == 0) {
        if (dr != 0) st_sys(p.ws[dr] + WS_FLAG + (rank * 32 + col * 8 + s8) * 4, call);
        st_agent(myws + WS_CALL + b * 4, call);
    }
    if (b == 0) {
        const u64 t0 = now();
        auto rf = __builtin_amdgcn_make_buffer_rsrc(myws + WS_FLAG, 0, NR * 128, 0x00020000);
        for (unsigned it = 1;; ++it) {
            const u32x4 v = __builtin_amdgcn_raw_buffer_load_b128(rf, lane * 16, 0, 17);
            const bool ok = (lane >> 3) == rank || (v.x >= call && v.y >= call && v.z >= call && v.w >= call);
            if (__all(ok)) break;
            if ((it & 255) == 0 && (ld_agent(myws + WS_TMO) == call || now() - t0 > DEADLINE_TICKS)) {
                st_agent(myws + WS_TMO, call);
                WAIT_VM(0);
                if (lane == 0) *(float*)(p.out[0] + (size_t)(rank * MLOC) * (NCOL * 4)) = __builtin_nanf("");
                break;
            }
        }
    }
}

extern "C" void kernel_launch(int rank, int nranks, ncclComm_t comm,
                              hipStream_t stream,
                              void* const* d_in, const int* in_sizes, int n_in,
                              void* d_out, int out_size,
                              const int* in_dtypes, int out_dtype,
                              void* const* d_out_all,
                              void* const* d_ws_all, size_t ws_bytes) {
    if (nranks != NR || (unsigned)rank >= (unsigned)NR || n_in < 4 || ws_bytes < WS_END ||
        in_sizes[0] != (int)X_BYTES || in_sizes[1] != (int)W_BYTES || out_size != 4096 * NCOL) {
        fprintf(stderr, "kernel_launch: unexpected problem shape\n");
        return;
    }
    HIP_CHECK(hipFuncSetAttribute((const void*)gemm_a2a, hipFuncAttributeMaxDynamicSharedMemorySize, LDS_BYTES));
    Params p;
    p.x = (const char*)d_in[0];
    p.w = (const char*)d_in[1];
    p.sx = (const float*)d_in[2];
    p.sw = (const float*)d_in[3];
    for (int i = 0; i < NR; ++i) { p.out[i] = (char*)d_out_all[(rank + i) % NR]; p.ws[i] = (char*)d_ws_all[(rank + i) % NR]; }
    p.dbg = nullptr;
    p.rank = rank;
    p.pad = 0;
    gemm_a2a<<<NWG, 768, LDS_BYTES, stream>>>(p);
    HIP_CHECK(hipGetLastError());
}
```

```cpp
#include <hip/hip_runtime.h>
#include <rccl/rccl.h>
#include <cstdio>
#include <cstdint>

#define HIP_CHECK(expr) do { hipError_t _e = (expr); if (_e != hipSuccess) { \
    fprintf(stderr, "HIP error %s at %s:%d\n", hipGetErrorString(_e), __FILE__, __LINE__); \
    return; } } while (0)

typedef int v2i __attribute__((ext_vector_type(2)));
typedef int v4i __attribute__((ext_vector_type(4)));
typedef unsigned u32x4 __attribute__((ext_vector_type(4)));
typedef float f32x4 __attribute__((ext_vector_type(4)));
typedef unsigned long long u64;

#define RBLK 8
#define SPLIT 0x1234455
#define DELAY 0x000159d
#define DELAY_L 14
#define INFL 3
#define PACE 912
#define LEAD 70
#define STAG_MIN 4
#define LEAD_MT 15
#define MAP2 1
#define EARLY_ACQ 0
#define PUSH0 70
#define NST 3
#define RES_AT 27
#define WARM 15
#define WARM_TILES 5

constexpr int NR = 8;
constexpr int KD = 4096;
constexpr int ND = 4096;
constexpr int MLOC = 512;
constexpr int NCOL = 512;
constexpr int RB = RBLK;
__host__ __device__ constexpr unsigned st_start(unsigned part) { return (part * RB + NST - 1) / NST; }
constexpr int NWG = 256;
constexpr unsigned OUT_BYTES = 4096u * NCOL * 4u;
constexpr unsigned X_BYTES = MLOC * KD;
constexpr unsigned W_BYTES = (unsigned)KD * ND;
constexpr unsigned WS_CALL = 0;
constexpr unsigned WS_TMO = 0x1000;
constexpr unsigned WS_FLAG = 0x2000;
constexpr unsigned WS_XFLAG = 0x3000;
constexpr unsigned WS_TT = 0x4000;
constexpr unsigned WS_XS = 0x10000;
constexpr unsigned XS_SLOT = RB * 16u * KD;
constexpr unsigned WS_END = WS_XS + NR * XS_SLOT;
constexpr u64 DEADLINE_TICKS = 100000000ull;

constexpr unsigned ST_X = 16384u;
constexpr unsigned STB = 28672u;
constexpr unsigned RING = 4u * STB;
constexpr unsigned LDS_T0 = RING;
constexpr unsigned LDS_DUMMY = RING + 256u;
constexpr unsigned LDS_BYTES = RING + 512u;
constexpr int NKT = KD / 128;

__host__ __device__ constexpr int split_mt(int s) { return (SPLIT >> (4 * (6 - s))) & 15; }
__host__ __device__ constexpr int split_start(int s) { int a = RB; for (int i = 0; i < s; ++i) a += split_mt(i); return a; }
__host__ __device__ constexpr u64 pack_starts() { u64 v = 0; for (int s = 0; s < 7; ++s) v |= (u64)split_start(s) << (8 * s); return v; }
static_assert(split_start(7) == 32, "remote split must cover the shard");
static_assert(RB >= 2 && RB <= 12, "RB");

struct Params {
    const char* x;
    const char* w;
    int rank;
    int pad;
    const float* sx;
    const float* sw;
    char* out[NR];
    char* ws[NR];
    u64* dbg;
};

__device__ __forceinline__ void st_sys(void* p, unsigned v) {
    __hip_atomic_store((unsigned*)p, v, __ATOMIC_RELAXED, __HIP_MEMORY_SCOPE_SYSTEM);
}
__device__ __forceinline__ unsigned ld_agent(const void* p) {
    return __builtin_amdgcn_readfirstlane(__hip_atomic_load((const unsigned*)p, __ATOMIC_RELAXED, __HIP_MEMORY_SCOPE_AGENT));
}
__device__ __forceinline__ void st_agent(void* p, unsigned v) {
    __hip_atomic_store((unsigned*)p, v, __ATOMIC_RELAXED, __HIP_MEMORY_SCOPE_AGENT);
}
__device__ __forceinline__ u64 now() { return __builtin_amdgcn_s_memrealtime(); }

#define LDS_AS __attribute__((address_space(3)))
typedef LDS_AS char lds_char;
__device__ __forceinline__ v4i lds_ld(unsigned a) { return *(const LDS_AS v4i*)(lds_char*)(uintptr_t)a; }
__device__ __forceinline__ void lds_st(unsigned a, v4i v) { *(LDS_AS v4i*)(lds_char*)(uintptr_t)a = v; }
__device__ __forceinline__ v2i lds_tr(unsigned a) { return __builtin_amdgcn_ds_read_tr8_b64_v2i32((LDS_AS v2i*)(lds_char*)(uintptr_t)a); }

#define BARRIER() do { asm volatile("" ::: "memory"); __builtin_amdgcn_s_barrier(); asm volatile("" ::: "memory"); } while (0)
#define WAIT_VM(n) asm volatile("s_waitcnt vmcnt(%0)" :: "i"(n) : "memory")
#define WAIT_LGKM0() do { asm volatile("" ::: "memory"); __builtin_amdgcn_s_waitcnt(0xC07F); asm volatile("" ::: "memory"); } while (0)
#define DMA16(m0v, voff, rsrc, soff) \
    asm volatile("s_mov_b32 m0, %0\n\ts_nop 0\n\tbuffer_load_dwordx4 %1, %2, %3 offen lds" \
                 :: "s"(m0v), "v"(voff), "s"(rsrc), "s"(soff) : "memory")
#define STORE16_SYS(data, voff, rsrc, soff) \
    asm volatile("buffer_store_dwordx4 %0, %1, %2, %3 offen sc0 sc1\n\ts_nop 1" :: "v"(data), "v"(voff), "s"(rsrc), "s"(soff) : "memory")
#define TOUCH(voff, rsrc) \
    asm volatile("s_mov_b32 m0, %0\n\ts_nop 0\n\tbuffer_load_dword %1, %2, 0 offen lds" :: "s"(ldsb + LDS_DUMMY), "v"(voff), "s"(rsrc) : "memory")

__device__ __forceinline__ void reserve(u64& slot, char* tt, u64 tmin, unsigned n) {
    const u64 a = tmin * 128u, d = (u64)n * PACE;
    asm volatile("global_atomic_umax_x2 %1, %2, %4\n\t"
                 "global_atomic_add_x2 %0, %1, %3, %4 sc0\n\ts_nop 1"
                 : "+v"(slot) : "v"(0u), "v"(a), "v"(d), "s"(tt) : "memory");
}
__device__ __forceinline__ void wait_until(unsigned due) {
    for (unsigned d = due - (unsigned)now(); d - 1u < 10000u; d = due - (unsigned)now()) __builtin_amdgcn_s_sleep(1);
}

#define FINE(k)

struct Ld {
    __amdgpu_buffer_rsrc_t rx, rw;
    unsigned wv[4], xv[3];
    unsigned a, c;
    unsigned kt;
};
template <int NX>
__device__ __forceinline__ void ld_stage(const Ld& l, unsigned sb, unsigned kt) {
    const unsigned a = l.a + sb, c = l.c + sb, s = kt * (128u * ND), t = kt * 128u;
    if (NX == 1)
        asm volatile("s_mov_b32 m0, %0\n\ts_nop 0\n\tbuffer_load_dwordx4 %1, %2, %3 offen lds"
                     :: "s"(c), "v"(l.xv[0]), "s"(l.rx), "s"(t) : "memory");
    if (NX == 2)
        asm volatile("s_mov_b32 m0, %0\n\ts_nop 0\n\tbuffer_load_dwordx4 %1, %3, %4 offen lds\n\t"
                     "buffer_load_dwordx4 %2, %3, %4 offen offset:1024 lds"
                     :: "s"(c), "v"(l.xv[0]), "v"(l.xv[1]), "s"(l.rx), "s"(t) : "memory");
    if (NX == 3)
        asm volatile("s_mov_b32 m0, %0\n\ts_nop 0\n\tbuffer_load_dwordx4 %1, %4, %5 offen lds\n\t"
                     "buffer_load_dwordx4 %2, %4, %5 offen offset:1024 lds\n\t"
                     "buffer_load_dwordx4 %3, %4, %5 offen offset:2048 lds"
                     :: "s"(c), "v"(l.xv[0]), "v"(l.xv[1]), "v"(l.xv[2]), "s"(l.rx), "s"(t) : "memory");
    asm volatile("s_mov_b32 m0, %0\n\ts_nop 0\n\tbuffer_load_dwordx4 %1, %5, %6 offen lds\n\t"
                 "buffer_load_dwordx4 %2, %5, %6 offen offset:1024 lds\n\t"
                 "buffer_load_dwordx4 %3, %5, %6 offen offset:2048 lds\n\t"
                 "buffer_load_dwordx4 %4, %5, %6 offen offset:3072 lds"
                 :: "s"(a), "v"(l.wv[0]), "v"(l.wv[1]), "v"(l.wv[2]), "v"(l.wv[3]), "s"(l.rw), "s"(s) : "memory");
}
template <int NX>
__device__ __forceinline__ void load_tile(const Ld& l) {
    constexpr int PER = 4 + NX;
    unsigned kt = l.kt;
    ld_stage<NX>(l, 0, kt); ld_stage<NX>(l, STB, kt + 1); ld_stage<NX>(l, 2 * STB, kt + 2);
    kt += 3;
    unsigned sb = 3 * STB;
#pragma unroll 1
    for (int n = 0; n < NKT - 3; ++n) {
        WAIT_VM(2 * PER); BARRIER();
        ld_stage<NX>(l, sb, kt);
        kt = (kt + 1) & (NKT - 1);
        sb = sb == 3 * STB ? 0 : sb + STB;
    }
    WAIT_VM(2 * PER); BARRIER();
    WAIT_VM(PER); BARRIER();
    WAIT_VM(0); BARRIER();
}

template <int MT>
struct Frags { v4i w[2], x[MT]; };

template <int MT>
__device__ __forceinline__ void mma_tile(unsigned ldsb, unsigned wid, unsigned lane, char* tt) {
    constexpr bool STAG = MT >= STAG_MIN;
    const unsigned cn = wid & 3, kh = wid >> 2;
    const unsigned lg = lane >> 4, m = lane & 15, q = m >> 1;
    unsigned wa[2];
#pragma unroll
    for (int t = 0; t < 2; ++t)
        wa[t] = ldsb + (8 * kh + 2 * lg) * 1024 + q * 128 + (((2 * cn + t) ^ ((q >> 1) | ((lg & 1) << 2))) * 16) + (m & 1) * 8;
    const unsigned xa = ldsb + ST_X + m * 128 + (((4 * kh + lg) ^ (m >> 1)) * 16);
    v4i acc[MT][2];
#pragma unroll
    for (int i = 0; i < MT; ++i) acc[i][0] = acc[i][1] = v4i{0, 0, 0, 0};
    auto rd = [&](Frags<MT>& f, unsigned sb) {
#pragma unroll
        for (int t = 0; t < 2; ++t) {
            const v2i lo = lds_tr(wa[t] + sb), hi = lds_tr(wa[t] + sb + 1024);
            f.w[t] = v4i{lo.x, lo.y, hi.x, hi.y};
        }
#pragma unroll
        for (int i = 0; i < MT; ++i) f.x[i] = lds_ld(xa + sb + i * 2048);
    };
    auto mm = [&](const Frags<MT>& f) {
#pragma unroll
        for (int i = 0; i < MT; ++i)
#pragma unroll
            for (int t = 0; t < 2; ++t)
                acc[i][t] = __builtin_amdgcn_mfma_i32_16x16x64_i8(f.w[t], f.x[i], acc[i][t], 0, 0, 0);
    };
    Frags<MT> fa, fb;
    u64 slot = 0;
    BARRIER();
    rd(fa, 0);
    if (STAG && kh) {
        unsigned sb = STB;
#pragma unroll 1
        for (int n = 1; n < NKT; ++n) {
            WAIT_LGKM0(); __builtin_amdgcn_sched_barrier(0);
            BARRIER();
            mm(fa);
            __builtin_amdgcn_sched_barrier(0);
            rd(fa, sb);
            __builtin_amdgcn_sched_barrier(0);
            sb = sb == 3 * STB ? 0 : sb + STB;
        }
        if (MT * 16384u > 3 * STB) {
            WAIT_LGKM0();
            BARRIER();
        }
        mm(fa);
    } else {
        unsigned sb = STB;
#pragma unroll 1
        for (int n = 1; n < NKT - 1; n += 2) {
            WAIT_LGKM0(); __builtin_amdgcn_sched_barrier(0);
            BARRIER();
            rd(fb, sb);
            __builtin_amdgcn_sched_barrier(0);
            mm(fa);
            __builtin_amdgcn_sched_barrier(0);
            sb = 4 * STB - sb;
            if (n == RES_AT && tt && lane == 0) reserve(slot, tt, now() + LEAD + (MT - 1) * LEAD_MT, MT * 8);
            WAIT_LGKM0(); __builtin_amdgcn_sched_barrier(0);
            BARRIER();
            rd(fa, sb - STB);
            __builtin_amdgcn_sched_barrier(0);
            mm(fb);
            __builtin_amdgcn_sched_barrier(0);
        }
        WAIT_LGKM0();
        BARRIER();
        rd(fb, 3 * STB);
        __builtin_amdgcn_sched_barrier(0);
        mm(fa);
        __builtin_amdgcn_sched_barrier(0);
        if (MT * 16384u > 3 * STB) {
            WAIT_LGKM0();
            BARRIER();
        }
        mm(fb);
    }
    const unsigned pw = ldsb + kh * (MT * 8192) + m * 512;
#pragma unroll
    for (int i = 0; i < MT; ++i)
#pragma unroll
        for (int t = 0; t < 2; ++t)
            lds_st(pw + i * 8192 + ((((2 * cn + t) * 4 + lg) ^ m) * 16), acc[i][t]);
    if (tt) {
        asm volatile("s_waitcnt vmcnt(0)" : "+v"(slot) :: "memory");
        if (lane == 0) *(volatile LDS_AS u64*)(lds_char*)(uintptr_t)(ldsb + LDS_T0) = slot;
    }
    WAIT_LGKM0();
    BARRIER();
}

__global__ void __launch_bounds__(768, 3) gemm_a2a(Params p) {
    extern __shared__ __attribute__((aligned(16))) char lds[];
    const u64 tstart = now();
    unsigned lane = threadIdx.x & 63;
    unsigned wid = __builtin_amdgcn_readfirstlane(threadIdx.x >> 6);
    const unsigned b = blockIdx.x;
    const unsigned rank = p.rank;
    const unsigned dr = ((b >> 2) & 1) | ((b >> 5) & 6);
    const unsigned lcol = b & 3;
    const unsigned col = (b - MAP2 * (dr >> 1)) & 3;
    const unsigned s8 = (b >> 3) & 7;
    const unsigned peer = (rank + dr) & (NR - 1);
    unsigned ldsb = (unsigned)(uintptr_t)(lds_char*)lds;
    auto rx = __builtin_amdgcn_make_buffer_rsrc((void*)p.x, 0, X_BYTES, 0x00020000);
    auto rw = __builtin_amdgcn_make_buffer_rsrc((void*)p.w, 0, W_BYTES, 0x00020000);
    const bool pusher = dr != 0 && s8 == 7;
    const unsigned dly = (dr != 0 ? (((unsigned)DELAY << 4) >> (4 * (7 - s8))) & 15 : DELAY_L) * 50u;

    const int njob = (dr != 0 && s8 < NST - 1) ? 2 : 1;

    if (wid >= 8) {
        const unsigned lw = wid - 8;
#pragma unroll 1
        for (int jb = 0; jb < njob; ++jb) {
            const bool staged = dr != 0 && (s8 == 7 || jb == 1);
            const bool remote = dr != 0 && !staged;
            unsigned mt, blk0;
            if (remote) { mt = ((unsigned)SPLIT >> (4 * (6 - s8))) & 15; blk0 = (unsigned)(pack_starts() >> (8 * s8)) & 255; }
            else if (staged) { const unsigned part = (s8 + 1) & 7; blk0 = st_start(part); mt = st_start(part + 1) - blk0; }
            else { mt = 4; blk0 = 4 * s8; }
            const unsigned l3 = lane >> 3, c8 = lane & 7;
            const unsigned nx = (2 * mt + 3 - lw) >> 2;
            const unsigned x0 = lw * (mt >> 1) + min(lw, (2 * mt) & 3);
            Ld l;
            l.rx = rx; l.rw = rw;
            if (staged) l.rx = __builtin_amdgcn_make_buffer_rsrc(p.ws[0] + WS_XS + peer * XS_SLOT, 0, XS_SLOT, 0x00020000);
            const unsigned v0 = l3 * KD + ((c8 ^ (l3 >> 1)) * 16);
            const unsigned wb = v0 + lw * 32u * ND + (remote ? peer * NCOL + col * 128 : rank * NCOL + lcol * 128);
#pragma unroll
            for (int k = 0; k < 4; ++k) l.wv[k] = (wb ^ ((k >> 1) * 64)) + k * (8u * ND - 1024u);
#pragma unroll
            for (int k = 0; k < 3; ++k) l.xv[k] = (v0 ^ (((x0 + k) & 1) * 64)) + (blk0 * 16u + x0 * 8u) * KD + k * (8u * KD - 1024u);
            l.a = ldsb + lw * 4096; l.c = ldsb + ST_X + x0 * 1024;
            l.kt = dr != 0 ? 8 * (s8 & 3) : 4 * s8;
            if (EARLY_ACQ && s8 == 7 && dr != 0 && lw == 0) {
                __builtin_amdgcn_fence(__ATOMIC_ACQUIRE, "");
                WAIT_VM(0);
            }
            if ((WARM & 32) && s8 == 7 && dr != 0) {
                while ((unsigned)(now() - tstart) < 500u) __builtin_amdgcn_s_sleep(8);
                for (unsigned e = 0; e < 4; ++e)
                    TOUCH(((dr >> 1) * 1024u + e * 256u + lw * 64u + lane) * ND + rank * NCOL + lcol * 128, rw);
            }
            if (staged || dly) BARRIER();
            FINE(1);
            if (nx == 1) load_tile<1>(l);
            else if (nx == 0) load_tile<0>(l);
            else if (nx == 2) load_tile<2>(l);
            else load_tile<3>(l);
            FINE(6);
            if (mt * 16384u > 3 * STB) BARRIER();
            BARRIER();
            FINE(7);
            if (jb + 1 < njob) BARRIER();
        }
        return;
    }

    if (EARLY_ACQ && wid == 7 && njob == 2) __builtin_amdgcn_fence(__ATOMIC_ACQUIRE, "");
    unsigned call = 0;
    if (pusher) {
        char* myws = p.ws[0];
        char* peerws = p.ws[dr];
        constexpr int NU = RB * 2;
        const unsigned u0 = (col * 8 * NU + wid) * 1024u;
        constexpr int D = 4;
        u32x4 pu[D];
#define LOAD_UNIT(k) asm volatile("buffer_load_dwordx4 %0, %1, %2, %3 offen" : "=v"(pu[(k) % D]) : "v"(lane * 16), "s"(rx), "s"(u0 + (k) * 8192u) : "memory")
#pragma unroll
        for (int k = 0; k < D; ++k) LOAD_UNIT(k);
        auto rs = __builtin_amdgcn_make_buffer_rsrc(peerws + WS_XS + rank * XS_SLOT, 0, XS_SLOT, 0x00020000);
        const u64 t0 = tstart + PUSH0;
        if (wid == 7 && lane == 0)
            __hip_atomic_fetch_max((u64*)(myws + WS_TT + (dr * 4 + col) * 128), t0 * 128u + 8u * NU * PACE, __ATOMIC_RELAXED, __HIP_MEMORY_SCOPE_AGENT);
#pragma unroll
        for (int k = 0; k < NU; ++k) {
            wait_until((unsigned)t0 + (((8 * k + wid) * PACE) >> 7));
            constexpr int dummy = 0; (void)dummy;
            const int yl = (k + D - 1 < NU ? D - 1 : NU - 1 - k), ys = (k < D - 1 ? k : D - 1);
            asm volatile("s_waitcnt vmcnt(%1)" : "+v"(pu[k % D]) : "i"(yl + ys) : "memory");
            STORE16_SYS(pu[k % D], lane * 16, rs, u0 + k * 8192u);
            if (k + D < NU) LOAD_UNIT(k + D);
        }
#undef LOAD_UNIT
        call = ld_agent(myws + WS_CALL + b * 4) + 1u;
        WAIT_VM(0);
        if (lane == 0) st_sys(peerws + WS_XFLAG + (rank * 32 + col * 8 + wid) * 4, call);
    } else if (dly) {
        const unsigned third = (dr - 1) >> 1;
        if ((WARM & 8) && (s8 == 5 || s8 == 6)) {
            const unsigned cb = (3u - (unsigned)((uintptr_t)p.w >> 7) - 4u * peer) & 7u;
            const unsigned ib = (b - cb) & 3, db = (dr & 1) | (ib << 1);
            if (cb < 4 && db != 0) {
                const unsigned t = threadIdx.x, h = (dr >> 1) * 2 + (s8 - 5);
                TOUCH(((t & 3) * 1024u + (t >> 2) * 8u + h) * ND + ((rank + db) & (NR - 1)) * NCOL + cb * 128, rw);
            }
        }
        if (dr != 0 && s8 >= 5) {
            if ((WARM & 1) && third < 3) {
                constexpr unsigned NE = (split_start(WARM_TILES) - RB) * 512u;
                for (unsigned e = (third * 2 + (s8 - 5)) * 512u + threadIdx.x; e - threadIdx.x < NE; e += 3072u)
                    TOUCH(e < NE ? RB * 16u * KD + e * 128u : ~0u, rx);
            }
            if (WARM & 2) {
                const unsigned o = ((s8 - 5) * 2048u + 512u + threadIdx.x) * ND + peer * NCOL + col * 128;
                TOUCH(o, rw);
                TOUCH(o + 1024u * ND, rw);
            }
            if ((WARM & 4) && third < 3) {
                const unsigned st = (unsigned)(pack_starts() >> (8 * s8)) & 255, n = (((unsigned)SPLIT >> (4 * (6 - s8))) & 15) * 512u;
                while ((unsigned)(now() - tstart) + 100u < dly) __builtin_amdgcn_s_sleep(8);
                const unsigned l0 = third * 512u + threadIdx.x, l1 = l0 + 1536u;
                TOUCH(l0 < n ? st * 16u * KD + l0 * 128u : ~0u, rx);
                TOUCH(l1 < n ? st * 16u * KD + l1 * 128u : ~0u, rx);
            }
        }
        while ((unsigned)(now() - tstart) < dly) __builtin_amdgcn_s_sleep(8);
    }

#pragma unroll 1
    for (int jb = 0; jb < njob; ++jb) {
        const bool staged = dr != 0 && (s8 == 7 || jb == 1);
        const bool remote = dr != 0 && !staged;
        unsigned mt, blk0;
        if (remote) { mt = ((unsigned)SPLIT >> (4 * (6 - s8))) & 15; blk0 = (unsigned)(pack_starts() >> (8 * s8)) & 255; }
        else if (staged) { const unsigned part = (s8 + 1) & 7; blk0 = st_start(part); mt = st_start(part + 1) - blk0; }
        else { mt = 4; blk0 = 4 * s8; }
        asm volatile("" : "+s"(ldsb), "+s"(wid), "+v"(lane));
        if (staged && wid == 0) {
            char* myws = p.ws[0];
            const u64 t0 = now();
            const char* f = myws + WS_XFLAG + (peer * 32 + (lane & 31)) * 4;
            for (unsigned it = 1;; ++it) {
                const unsigned v = __hip_atomic_load((const unsigned*)f, __ATOMIC_RELAXED, __HIP_MEMORY_SCOPE_SYSTEM);
                if (__all(v >= call)) break;
                if ((it & 63) == 0 && (ld_agent(myws + WS_TMO) == call || now() - t0 > DEADLINE_TICKS)) {
                    st_agent(myws + WS_TMO, call);
                    break;
                }
            }
            if (!EARLY_ACQ) __builtin_amdgcn_fence(__ATOMIC_ACQUIRE, "");
        }
        if (staged) WAIT_VM(0);
        if (staged || dly) BARRIER();
        if (staged && (WARM & 16)) {
            auto rs = __builtin_amdgcn_make_buffer_rsrc(p.ws[0] + WS_XS + peer * XS_SLOT, 0, XS_SLOT, 0x00020000);
            for (unsigned e = 0; e < mt; ++e) TOUCH((blk0 + e) * 16u * KD + (wid * 64 + lane) * 128u, rs);
        }
        char* tt = (remote && wid == 0) ? p.ws[0] + WS_TT + (dr * 4 + col) * 128 : nullptr;
        switch (mt) {
            case 1: mma_tile<1>(ldsb, wid, lane, tt); break;
            case 2: mma_tile<2>(ldsb, wid, lane, tt); break;
            case 3: mma_tile<3>(ldsb, wid, lane, tt); break;
            case 4: mma_tile<4>(ldsb, wid, lane, tt); break;
            case 5: mma_tile<5>(ldsb, wid, lane, tt); break;
            default: mma_tile<6>(ldsb, wid, lane, tt); break;
        }
        typedef const __attribute__((address_space(4))) float* cfp;
        const float scale = *(cfp)p.sx * *(cfp)p.sw;
        auto ro = __builtin_amdgcn_make_buffer_rsrc(remote ? p.out[dr] : p.out[0], 0, OUT_BYTES, 0x00020000);
        const unsigned orow = (staged ? peer : rank) * MLOC + blk0 * 16u;
        const unsigned rl = lane >> 5, ch = lane & 31;
        const unsigned ovoff = rl * (NCOL * 4) + (remote ? col : lcol) * 512 + ch * 16;
        const bool bad = staged && ld_agent(p.ws[0] + WS_TMO) == call;
        unsigned t0 = 0;
        if (remote) t0 = __builtin_amdgcn_readfirstlane((unsigned)(*(volatile LDS_AS u64*)(lds_char*)(uintptr_t)(ldsb + LDS_T0) >> 7));
#pragma unroll 1
        for (unsigned k = wid; k < mt * 8; k += 8) {
            const unsigned r = 2 * k + rl;
            const unsigned a = ldsb + r * 512 + ((ch ^ (r & 15)) * 16);
            const v4i v = lds_ld(a) + lds_ld(a + mt * 8192u);
            f32x4 f;
#pragma unroll
            for (int e = 0; e < 4; ++e) f[e] = fmaxf((float)v[e] * scale, 0.0f);
            if (bad) f = f32x4{__builtin_nanf(""), __builtin_nanf(""), __builtin_nanf(""), __builtin_nanf("")};
            if (remote) {
                asm volatile("" : "+v"(f));
                wait_until(t0 + ((k * PACE) >> 7));
                WAIT_VM(INFL - 1);
            }
            STORE16_SYS(f, ovoff, ro, (orow + 2 * k) * (NCOL * 4));
        }
        if (!staged && (wid == 0 || njob == 2)) call = ld_agent(p.ws[0] + WS_CALL + b * 4) + 1u;
        if (remote) WAIT_VM(0);
        BARRIER();
    }
    if (wid != 0) return;
    char* myws = p.ws[0];
    if (lane == 0) {
        if (dr != 0) st_sys(p.ws[dr] + WS_FLAG + (rank * 32 + col * 8 + s8) * 4, call);
        st_agent(myws + WS_CALL + b * 4, call);
    }
    if (b == 0) {
        const u64 t0 = now();
        auto rf = __builtin_amdgcn_make_buffer_rsrc(myws + WS_FLAG, 0, NR * 128, 0x00020000);
        for (unsigned it = 1;; ++it) {
            const u32x4 v = __builtin_amdgcn_raw_buffer_load_b128(rf, lane * 16, 0, 17);
            const bool ok = (lane >> 3) == rank || (v.x >= call && v.y >= call && v.z >= call && v.w >= call);
            if (__all(ok)) break;
            if ((it & 255) == 0 && (ld_agent(myws + WS_TMO) == call || now() - t0 > DEADLINE_TICKS)) {
                st_agent(myws + WS_TMO, call);
                WAIT_VM(0);
                if (lane == 0) *(float*)(p.out[0] + (size_t)(rank * MLOC) * (NCOL * 4)) = __builtin_nanf("");
                break;
            }
        }
    }
}

extern "C" void kernel_launch(int rank, int nranks, ncclComm_t comm,
                              hipStream_t stream,
                              void* const* d_in, const int* in_sizes, int n_in,
                              void* d_out, int out_size,
                              const int* in_dtypes, int out_dtype,
                              void* const* d_out_all,
                              void* const* d_ws_all, size_t ws_bytes) {
    if (nranks != NR || (unsigned)rank >= (unsigned)NR || n_in < 4 || ws_bytes < WS_END ||
        in_sizes[0] != (int)X_BYTES || in_sizes[1] != (int)W_BYTES || out_size != 4096 * NCOL) {
        fprintf(stderr, "kernel_launch: unexpected problem shape\n");
        return;
    }
    HIP_CHECK(hipFuncSetAttribute((const void*)gemm_a2a, hipFuncAttributeMaxDynamicSharedMemorySize, LDS_BYTES));
    Params p;
    p.x = (const char*)d_in[0];
    p.w = (const char*)d_in[1];
    p.sx = (const float*)d_in[2];
    p.sw = (const float*)d_in[3];
    for (int i = 0; i < NR; ++i) { p.out[i] = (char*)d_out_all[(rank + i) % NR]; p.ws[i] = (char*)d_ws_all[(rank + i) % NR]; }
    p.dbg = nullptr;
    p.rank = rank;
    p.pad = 0;
    gemm_a2a<<<NWG, 768, LDS_BYTES, stream>>>(p);
    HIP_CHECK(hipGetLastError());
}
```

```cpp
#include <hip/hip_runtime.h>
#include <rccl/rccl.h>
#include <cstdio>
#include <cstdint>

#define HIP_CHECK(expr) do { hipError_t _e = (expr); if (_e != hipSuccess) { \
    fprintf(stderr, "HIP error %s at %s:%d\n", hipGetErrorString(_e), __FILE__, __LINE__); \
    return; } } while (0)

typedef int v2i __attribute__((ext_vector_type(2)));
typedef int v4i __attribute__((ext_vector_type(4)));
typedef unsigned u32x4 __attribute__((ext_vector_type(4)));
typedef float f32x4 __attribute__((ext_vector_type(4)));
typedef unsigned long long u64;

#define RBLK 7
#define SPLIT 0x1235455
#define DELAY 0x00008af
#define DELAY_L 14
#define INFL 3
#define PACE 888
#define LEAD 70
#define STAG_MIN 4
#define LEAD_MT 15
#define MAP2 1
#define PUSH0 55
#define NST 3
#define ROT 0
#define WARM_K0 2
#define RES_AT 27
#define WARM 68
#define WARM_TILES 5

constexpr int NR = 8;
constexpr int KD = 4096;
constexpr int ND = 4096;
constexpr int MLOC = 512;
constexpr int NCOL = 512;
constexpr int RB = RBLK;
__host__ __device__ constexpr unsigned st_start(unsigned part) { return (part * RB + NST - 1) / NST; }
constexpr int NWG = 256;
constexpr unsigned OUT_BYTES = 4096u * NCOL * 4u;
constexpr unsigned X_BYTES = MLOC * KD;
constexpr unsigned W_BYTES = (unsigned)KD * ND;
constexpr unsigned WS_CALL = 0;
constexpr unsigned WS_TMO = 0x1000;
constexpr unsigned WS_FLAG = 0x2000;
constexpr unsigned WS_XFLAG = 0x3000;
constexpr unsigned WS_TT = 0x4000;
constexpr unsigned WS_XS = 0x10000;
constexpr unsigned XS_SLOT = RB * 16u * KD;
constexpr unsigned WS_END = WS_XS + NR * XS_SLOT;
constexpr u64 DEADLINE_TICKS = 100000000ull;

constexpr unsigned ST_X = 16384u;
constexpr unsigned STB = 28672u;
constexpr unsigned RING = 4u * STB;
constexpr unsigned LDS_T0 = RING;
constexpr unsigned LDS_DUMMY = RING + 256u;
constexpr unsigned LDS_BYTES = RING + 512u;
constexpr int NKT = KD / 128;

__host__ __device__ constexpr int split_mt(int s) { return (SPLIT >> (4 * (6 - s))) & 15; }
__host__ __device__ constexpr int split_start(int s) { int a = RB; for (int i = 0; i < s; ++i) a += split_mt(i); return a; }
__host__ __device__ constexpr u64 pack_starts() { u64 v = 0; for (int s = 0; s < 7; ++s) v |= (u64)split_start(s) << (8 * s); return v; }
static_assert(split_start(7) == 32, "remote split must cover the shard");
static_assert(RB >= 2 && RB <= 12, "RB");

struct Params {
    const char* x;
    const char* w;
    char* xs;
    int rank;
    int pad;
    const float* sx;
    const float* sw;
    char* out[NR];
    char* ws[NR];
    u64* dbg;
};

__device__ __forceinline__ void st_sys(void* p, unsigned v) {
    __hip_atomic_store((unsigned*)p, v, __ATOMIC_RELAXED, __HIP_MEMORY_SCOPE_SYSTEM);
}
__device__ __forceinline__ unsigned ld_agent(const void* p) {
    return __builtin_amdgcn_readfirstlane(__hip_atomic_load((const unsigned*)p, __ATOMIC_RELAXED, __HIP_MEMORY_SCOPE_AGENT));
}
__device__ __forceinline__ void st_agent(void* p, unsigned v) {
    __hip_atomic_store((unsigned*)p, v, __ATOMIC_RELAXED, __HIP_MEMORY_SCOPE_AGENT);
}
__device__ __forceinline__ u64 now() { return __builtin_amdgcn_s_memrealtime(); }

#define LDS_AS __attribute__((address_space(3)))
typedef LDS_AS char lds_char;
__device__ __forceinline__ v4i lds_ld(unsigned a) { return *(const LDS_AS v4i*)(lds_char*)(uintptr_t)a; }
__device__ __forceinline__ void lds_st(unsigned a, v4i v) { *(LDS_AS v4i*)(lds_char*)(uintptr_t)a = v; }
__device__ __forceinline__ v2i lds_tr(unsigned a) { return __builtin_amdgcn_ds_read_tr8_b64_v2i32((LDS_AS v2i*)(lds_char*)(uintptr_t)a); }

#define BARRIER() do { asm volatile("" ::: "memory"); __builtin_amdgcn_s_barrier(); asm volatile("" ::: "memory"); } while (0)
#define WAIT_VM(n) asm volatile("s_waitcnt vmcnt(%0)" :: "i"(n) : "memory")
#define WAIT_LGKM0() do { asm volatile("" ::: "memory"); __builtin_amdgcn_s_waitcnt(0xC07F); asm volatile("" ::: "memory"); } while (0)
#define DMA16(m0v, voff, rsrc, soff) \
    asm volatile("s_mov_b32 m0, %0\n\ts_nop 0\n\tbuffer_load_dwordx4 %1, %2, %3 offen lds" \
                 :: "s"(m0v), "v"(voff), "s"(rsrc), "s"(soff) : "memory")
#define STORE16_SYS(data, voff, rsrc, soff) \
    asm volatile("buffer_store_dwordx4 %0, %1, %2, %3 offen sc0 sc1\n\ts_nop 1" :: "v"(data), "v"(voff), "s"(rsrc), "s"(soff) : "memory")
#define TOUCH(voff, rsrc) \
    asm volatile("s_mov_b32 m0, %0\n\ts_nop 0\n\tbuffer_load_dword %1, %2, 0 offen lds" :: "s"(ldsb + LDS_DUMMY), "v"(voff), "s"(rsrc) : "memory")

__device__ __forceinline__ void reserve(u64& slot, char* tt, u64 tmin, unsigned n) {
    const u64 a = tmin * 128u, d = (u64)n * PACE;
    asm volatile("global_atomic_umax_x2 %1, %2, %4\n\t"
                 "global_atomic_add_x2 %0, %1, %3, %4 sc0\n\ts_nop 1"
                 : "+v"(slot) : "v"(0u), "v"(a), "v"(d), "s"(tt) : "memory");
}
__device__ __forceinline__ void wait_until(unsigned due) {
    for (unsigned d = due - (unsigned)now(); d - 1u < 10000u; d = due - (unsigned)now()) __builtin_amdgcn_s_sleep(1);
}

#define FINE(k)

struct Ld {
    __amdgpu_buffer_rsrc_t rx, rw;
    unsigned wv[4], xv[3];
    unsigned a, c;
    unsigned kt;
    unsigned wcol;
};
template <int NX>
__device__ __forceinline__ void ld_stage(const Ld& l, unsigned sb, unsigned kt) {
    const unsigned a = l.a + sb, c = l.c + sb, s = kt * (128u * ND) + l.wcol, t = kt * 128u;
    if (NX == 1)
        asm volatile("s_mov_b32 m0, %0\n\ts_nop 0\n\tbuffer_load_dwordx4 %1, %2, %3 offen lds"
                     :: "s"(c), "v"(l.xv[0]), "s"(l.rx), "s"(t) : "memory");
    if (NX == 2)
        asm volatile("s_mov_b32 m0, %0\n\ts_nop 0\n\tbuffer_load_dwordx4 %1, %3, %4 offen lds\n\t"
                     "buffer_load_dwordx4 %2, %3, %4 offen offset:1024 lds"
                     :: "s"(c), "v"(l.xv[0]), "v"(l.xv[1]), "s"(l.rx), "s"(t) : "memory");
    if (NX == 3)
        asm volatile("s_mov_b32 m0, %0\n\ts_nop 0\n\tbuffer_load_dwordx4 %1, %4, %5 offen lds\n\t"
                     "buffer_load_dwordx4 %2, %4, %5 offen offset:1024 lds\n\t"
                     "buffer_load_dwordx4 %3, %4, %5 offen offset:2048 lds"
                     :: "s"(c), "v"(l.xv[0]), "v"(l.xv[1]), "v"(l.xv[2]), "s"(l.rx), "s"(t) : "memory");
    asm volatile("s_mov_b32 m0, %0\n\ts_nop 0\n\tbuffer_load_dwordx4 %1, %5, %6 offen lds\n\t"
                 "buffer_load_dwordx4 %2, %5, %6 offen offset:1024 lds\n\t"
                 "buffer_load_dwordx4 %3, %5, %6 offen offset:2048 lds\n\t"
                 "buffer_load_dwordx4 %4, %5, %6 offen offset:3072 lds"
                 :: "s"(a), "v"(l.wv[0]), "v"(l.wv[1]), "v"(l.wv[2]), "v"(l.wv[3]), "s"(l.rw), "s"(s) : "memory");
}
template <int NX>
__device__ __forceinline__ void load_tile(const Ld& l) {
    constexpr int PER = 4 + NX;
    unsigned kt = l.kt;
    ld_stage<NX>(l, 0, kt); ld_stage<NX>(l, STB, kt + 1); ld_stage<NX>(l, 2 * STB, kt + 2);
    kt += 3;
    unsigned sb = 3 * STB;
#pragma unroll 1
    for (int n = 0; n < NKT - 3; ++n) {
        WAIT_VM(2 * PER); BARRIER();
        ld_stage<NX>(l, sb, kt);
        kt = (kt + 1) & (NKT - 1);
        sb = sb == 3 * STB ? 0 : sb + STB;
    }
    WAIT_VM(2 * PER); BARRIER();
    WAIT_VM(PER); BARRIER();
    WAIT_VM(0); BARRIER();
}

template <int MT>
struct Frags { v4i w[2], x[MT]; };

template <int MT>
__device__ __forceinline__ void mma_tile(unsigned ldsb, unsigned wid, unsigned lane, char* tt) {
    constexpr bool STAG = MT >= STAG_MIN;
    const unsigned cn = wid & 3, kh = wid >> 2;
    const unsigned lg = lane >> 4, m = lane & 15, q = m >> 1;
    unsigned wa[2];
#pragma unroll
    for (int t = 0; t < 2; ++t)
        wa[t] = ldsb + (8 * kh + 2 * lg) * 1024 + q * 128 + (((2 * cn + t) ^ ((q >> 1) | ((lg & 1) << 2))) * 16) + (m & 1) * 8;
    const unsigned xa = ldsb + ST_X + m * 128 + (((4 * kh + lg) ^ (m >> 1)) * 16);
    v4i acc[MT][2];
#pragma unroll
    for (int i = 0; i < MT; ++i) acc[i][0] = acc[i][1] = v4i{0, 0, 0, 0};
    auto rd = [&](Frags<MT>& f, unsigned sb) {
#pragma unroll
        for (int t = 0; t < 2; ++t) {
            const v2i lo = lds_tr(wa[t] + sb), hi = lds_tr(wa[t] + sb + 1024);
            f.w[t] = v4i{lo.x, lo.y, hi.x, hi.y};
        }
#pragma unroll
        for (int i = 0; i < MT; ++i) f.x[i] = lds_ld(xa + sb + i * 2048);
    };
    auto mm = [&](const Frags<MT>& f) {
#pragma unroll
        for (int i = 0; i < MT; ++i)
#pragma unroll
            for (int t = 0; t < 2; ++t)
                acc[i][t] = __builtin_amdgcn_mfma_i32_16x16x64_i8(f.w[t], f.x[i], acc[i][t], 0, 0, 0);
    };
    Frags<MT> fa, fb;
    u64 slot = 0;
    BARRIER();
    rd(fa, 0);
    if (STAG && kh) {
        unsigned sb = STB;
#pragma unroll 1
        for (int n = 1; n < NKT; ++n) {
            WAIT_LGKM0(); __builtin_amdgcn_sched_barrier(0);
            BARRIER();
            mm(fa);
            __builtin_amdgcn_sched_barrier(0);
            rd(fa, sb);
            __builtin_amdgcn_sched_barrier(0);
            sb = sb == 3 * STB ? 0 : sb + STB;
        }
        if (MT * 16384u > 3 * STB) {
            WAIT_LGKM0();
            BARRIER();
        }
        mm(fa);
    } else {
        unsigned sb = STB;
#pragma unroll 1
        for (int n = 1; n < NKT - 1; n += 2) {
            WAIT_LGKM0(); __builtin_amdgcn_sched_barrier(0);
            BARRIER();
            rd(fb, sb);
            __builtin_amdgcn_sched_barrier(0);
            mm(fa);
            __builtin_amdgcn_sched_barrier(0);
            sb = 4 * STB - sb;
            if (n == RES_AT && tt && lane == 0) reserve(slot, tt, now() + LEAD + (MT - 1) * LEAD_MT, MT * 8);
            WAIT_LGKM0(); __builtin_amdgcn_sched_barrier(0);
            BARRIER();
            rd(fa, sb - STB);
            __builtin_amdgcn_sched_barrier(0);
            mm(fb);
            __builtin_amdgcn_sched_barrier(0);
        }
        WAIT_LGKM0();
        BARRIER();
        rd(fb, 3 * STB);
        __builtin_amdgcn_sched_barrier(0);
        mm(fa);
        __builtin_amdgcn_sched_barrier(0);
        if (MT * 16384u > 3 * STB) {
            WAIT_LGKM0();
            BARRIER();
        }
        mm(fb);
    }
    const unsigned pw = ldsb + kh * (MT * 8192) + m * 512;
#pragma unroll
    for (int i = 0; i < MT; ++i)
#pragma unroll
        for (int t = 0; t < 2; ++t)
            lds_st(pw + i * 8192 + ((((2 * cn + t) * 4 + lg) ^ m) * 16), acc[i][t]);
    if (tt) {
        asm volatile("s_waitcnt vmcnt(0)" : "+v"(slot) :: "memory");
        if (lane == 0) *(volatile LDS_AS u64*)(lds_char*)(uintptr_t)(ldsb + LDS_T0) = slot;
    }
    WAIT_LGKM0();
    BARRIER();
}

__global__ void __launch_bounds__(768, 3) gemm_a2a(Params p) {
    extern __shared__ __attribute__((aligned(16))) char lds[];
    asm volatile("s_load_dwordx8 s[92:99], %0, 0x0" :: "s"(__builtin_amdgcn_kernarg_segment_ptr())
                 : "s92", "s93", "s94", "s95", "s96", "s97", "s98", "s99");
    char *kx, *kw, *kxs;
    unsigned rank;
#define KA_GET() asm volatile("s_waitcnt lgkmcnt(0)\n\ts_mov_b64 %0, s[92:93]\n\ts_mov_b64 %1, s[94:95]\n\t" \
                              "s_mov_b64 %2, s[96:97]\n\ts_mov_b32 %3, s98" : "=s"(kx), "=s"(kw), "=s"(kxs), "=s"(rank))
    unsigned lane = threadIdx.x & 63;
    unsigned wid = __builtin_amdgcn_readfirstlane(threadIdx.x >> 6) - 4;
    const unsigned b = blockIdx.x;
    const unsigned dr = ((b >> 2) & 1) | ((b >> 5) & 6);
    const unsigned lcol = b & 3;
    const unsigned col = (b - MAP2 * (dr >> 1)) & 3;
    const unsigned s8 = (b >> 3) & 7;
    unsigned ldsb = (unsigned)(uintptr_t)(lds_char*)lds;
    const bool pusher = dr != 0 && s8 == 7;
    const unsigned dly = (dr != 0 ? (((unsigned)DELAY << 4) >> (4 * (7 - s8))) & 15 : DELAY_L) * 50u;
    const int njob = (dr != 0 && s8 < NST - 1) ? 2 : 1;

    if (wid >= 8) {
        const unsigned lw = wid + 4;
        const unsigned l3 = lane >> 3, c8 = lane & 7;
        const unsigned v0 = l3 * KD + ((c8 ^ (l3 >> 1)) * 16);
        Ld l;
#pragma unroll
        for (int k = 0; k < 4; ++k) l.wv[k] = (v0 ^ ((k >> 1) * 64)) + lw * 32u * ND + k * (8u * ND - 1024u);
        l.a = ldsb + lw * 4096;
#pragma unroll 1
        for (int jb = 0; jb < njob; ++jb) {
            const bool staged = dr != 0 && (s8 == 7 || jb == 1);
            const bool remote = dr != 0 && !staged;
            unsigned mt, blk0;
            if (remote) { mt = ((unsigned)SPLIT >> (4 * (6 - s8))) & 15; blk0 = (unsigned)(pack_starts() >> (8 * s8)) & 255; }
            else if (staged) { const unsigned part = (s8 + 1) & 7; blk0 = st_start(part); mt = st_start(part + 1) - blk0; }
            else { mt = 4; blk0 = 4 * s8; }
            const unsigned nx = (2 * mt + 3 - lw) >> 2;
            const unsigned x0 = lw * (mt >> 1) + min(lw, (2 * mt) & 3);
#pragma unroll
            for (int k = 0; k < 3; ++k) l.xv[k] = (v0 ^ (((x0 + k) & 1) * 64)) + (blk0 * 16u + x0 * 8u) * KD + k * (8u * KD - 1024u);
            l.c = ldsb + ST_X + x0 * 1024;
            l.kt = remote ? ROT * (s8 & 3) : dr != 0 ? 8 * (dr >> 1) : 4 * s8;
            if (staged || dly) BARRIER();
            asm volatile("" : "+v"(l.xv[0]), "+v"(l.xv[1]), "+v"(l.xv[2]), "+s"(l.c), "+s"(l.kt));
            KA_GET();
            const unsigned peer = (rank + dr) & (NR - 1);
            l.wcol = remote ? peer * NCOL + col * 128 : rank * NCOL + lcol * 128;
            l.rw = __builtin_amdgcn_make_buffer_rsrc(kw, 0, W_BYTES, 0x00020000);
            l.rx = __builtin_amdgcn_make_buffer_rsrc(staged ? kxs + peer * XS_SLOT : kx, 0, staged ? XS_SLOT : X_BYTES, 0x00020000);
            FINE(1);
            if (nx == 1) load_tile<1>(l);
            else if (nx == 0) load_tile<0>(l);
            else if (nx == 2) load_tile<2>(l);
            else load_tile<3>(l);
            FINE(6);
            if (mt * 16384u > 3 * STB) BARRIER();
            BARRIER();
            FINE(7);
            if (jb + 1 < njob) BARRIER();
        }
        __builtin_amdgcn_endpgm();
    }

    const u64 tstart = now();
    KA_GET();
    const unsigned peer = (rank + dr) & (NR - 1);
    const unsigned tid = threadIdx.x - 256;
    auto rx = __builtin_amdgcn_make_buffer_rsrc(kx, 0, X_BYTES, 0x00020000);
    auto rw = __builtin_amdgcn_make_buffer_rsrc(kw, 0, W_BYTES, 0x00020000);
    unsigned call = 0;
    if (pusher) {
        char* myws = p.ws[0];
        char* peerws = p.ws[dr];
        constexpr int NU = RB * 2;
        const unsigned u0 = (col * 8 * NU + wid) * 1024u;
        constexpr int D = 4;
        u32x4 pu[D];
#define LOAD_UNIT(k) asm volatile("buffer_load_dwordx4 %0, %1, %2, %3 offen" : "=v"(pu[(k) % D]) : "v"(lane * 16), "s"(rx), "s"(u0 + (k) * 8192u) : "memory")
#pragma unroll
        for (int k = 0; k < D; ++k) LOAD_UNIT(k);
        auto rs = __builtin_amdgcn_make_buffer_rsrc(peerws + WS_XS + rank * XS_SLOT, 0, XS_SLOT, 0x00020000);
        const u64 t0 = tstart + PUSH0;
        if (wid == 7 && lane == 0)
            __hip_atomic_fetch_max((u64*)(myws + WS_TT + (dr * 4 + col) * 128), t0 * 128u + 8u * NU * PACE, __ATOMIC_RELAXED, __HIP_MEMORY_SCOPE_AGENT);
#pragma unroll
        for (int k = 0; k < NU; ++k) {
            wait_until((unsigned)t0 + (((8 * k + wid) * PACE) >> 7));
            constexpr int dummy = 0; (void)dummy;
            const int yl = (k + D - 1 < NU ? D - 1 : NU - 1 - k), ys = (k < D - 1 ? k : D - 1);
            asm volatile("s_waitcnt vmcnt(%1)" : "+v"(pu[k % D]) : "i"(yl + ys) : "memory");
            STORE16_SYS(pu[k % D], lane * 16, rs, u0 + k * 8192u);
            if (k + D < NU) LOAD_UNIT(k + D);
        }
#undef LOAD_UNIT
        call = ld_agent(myws + WS_CALL + b * 4) + 1u;
        WAIT_VM(0);
        if (lane == 0) st_sys(peerws + WS_XFLAG + (rank * 32 + col * 8 + wid) * 4, call);
    } else if (dly) {
        const unsigned third = (dr - 1) >> 1;
        if ((WARM & 64) && s8 >= 4 && s8 < 7) {
            const unsigned h = (dr >> 1) * 3 + (s8 - 4);
#pragma unroll 1
            for (unsigned kt = wid; kt < NKT; kt += 8) {
                const unsigned item = (h + kt) % 12u;
                if (kt < WARM_K0) continue;
                if (item < 8) {
                    const unsigned i = item >> 1, d = (dr & 1) | (i << 1);
                    if (d != 0) TOUCH((kt * 128u + (item & 1) * 64u + lane) * ND + ((rank + d) & (NR - 1)) * NCOL + ((b - MAP2 * i) & 3) * 128, rw);
                } else {
                    const unsigned r = RB * 16u + (item - 8) * 64u + lane;
                    TOUCH(r < split_start(WARM_TILES) * 16u ? r * KD + kt * 128u : ~0u, rx);
                }
            }
        }
        if (dr != 0 && s8 >= 5) {
            if ((WARM & 4) && third < 3) {
                const unsigned st = (unsigned)(pack_starts() >> (8 * s8)) & 255, n = (((unsigned)SPLIT >> (4 * (6 - s8))) & 15) * 512u;
                while ((unsigned)(now() - tstart) + 100u < dly) __builtin_amdgcn_s_sleep(8);
                const unsigned l0 = third * 512u + tid, l1 = l0 + 1536u;
                TOUCH(l0 < n ? st * 16u * KD + l0 * 128u : ~0u, rx);
                TOUCH(l1 < n ? st * 16u * KD + l1 * 128u : ~0u, rx);
            }
        }
        while ((unsigned)(now() - tstart) < dly) __builtin_amdgcn_s_sleep(8);
    }

#pragma unroll 1
    for (int jb = 0; jb < njob; ++jb) {
        const bool staged = dr != 0 && (s8 == 7 || jb == 1);
        const bool remote = dr != 0 && !staged;
        unsigned mt, blk0;
        if (remote) { mt = ((unsigned)SPLIT >> (4 * (6 - s8))) & 15; blk0 = (unsigned)(pack_starts() >> (8 * s8)) & 255; }
        else if (staged) { const unsigned part = (s8 + 1) & 7; blk0 = st_start(part); mt = st_start(part + 1) - blk0; }
        else { mt = 4; blk0 = 4 * s8; }
        asm volatile("" : "+s"(ldsb), "+s"(wid), "+v"(lane));
        if (staged && wid == 0) {
            char* myws = p.ws[0];
            const u64 t0 = now();
            const char* f = myws + WS_XFLAG + (peer * 32 + (lane & 31)) * 4;
            for (unsigned it = 1;; ++it) {
                const unsigned v = __hip_atomic_load((const unsigned*)f, __ATOMIC_RELAXED, __HIP_MEMORY_SCOPE_SYSTEM);
                if (__all(v >= call)) break;
                if ((it & 63) == 0 && (ld_agent(myws + WS_TMO) == call || now() - t0 > DEADLINE_TICKS)) {
                    st_agent(myws + WS_TMO, call);
                    break;
                }
            }
            __builtin_amdgcn_fence(__ATOMIC_ACQUIRE, "");
            WAIT_VM(0);
        }
        if (staged || dly) BARRIER();
        char* tt = (remote && wid == 0) ? p.ws[0] + WS_TT + (dr * 4 + col) * 128 : nullptr;
        switch (mt) {
            case 1: mma_tile<1>(ldsb, wid, lane, tt); break;
            case 2: mma_tile<2>(ldsb, wid, lane, tt); break;
            case 3: mma_tile<3>(ldsb, wid, lane, tt); break;
            case 4: mma_tile<4>(ldsb, wid, lane, tt); break;
            case 5: mma_tile<5>(ldsb, wid, lane, tt); break;
            default: mma_tile<6>(ldsb, wid, lane, tt); break;
        }
        typedef const __attribute__((address_space(4))) float* cfp;
        const float scale = *(cfp)p.sx * *(cfp)p.sw;
        auto ro = __builtin_amdgcn_make_buffer_rsrc(remote ? p.out[dr] : p.out[0], 0, OUT_BYTES, 0x00020000);
        const unsigned orow = (staged ? peer : rank) * MLOC + blk0 * 16u;
        const unsigned rl = lane >> 5, ch = lane & 31;
        const unsigned ovoff = rl * (NCOL * 4) + (remote ? col : lcol) * 512 + ch * 16;
        const bool bad = staged && ld_agent(p.ws[0] + WS_TMO) == call;
        unsigned t0 = 0;
        if (remote) t0 = __builtin_amdgcn_readfirstlane((unsigned)(*(volatile LDS_AS u64*)(lds_char*)(uintptr_t)(ldsb + LDS_T0) >> 7));
#pragma unroll 1
        for (unsigned k = wid; k < mt * 8; k += 8) {
            const unsigned r = 2 * k + rl;
            const unsigned a = ldsb + r * 512 + ((ch ^ (r & 15)) * 16);
            const v4i v = lds_ld(a) + lds_ld(a + mt * 8192u);
            f32x4 f;
#pragma unroll
            for (int e = 0; e < 4; ++e) f[e] = fmaxf((float)v[e] * scale, 0.0f);
            if (bad) f = f32x4{__builtin_nanf(""), __builtin_nanf(""), __builtin_nanf(""), __builtin_nanf("")};
            if (remote) {
                asm volatile("" : "+v"(f));
                wait_until(t0 + ((k * PACE) >> 7));
                WAIT_VM(INFL - 1);
            }
            STORE16_SYS(f, ovoff, ro, (orow + 2 * k) * (NCOL * 4));
        }
        if (!staged && (wid == 0 || njob == 2)) call = ld_agent(p.ws[0] + WS_CALL + b * 4) + 1u;
        if (remote) WAIT_VM(0);
        BARRIER();
    }
    if (wid != 0) return;
    char* myws = p.ws[0];
    if (lane == 0) {
        if (dr != 0) st_sys(p.ws[dr] + WS_FLAG + (rank * 32 + col * 8 + s8) * 4, call);
        st_agent(myws + WS_CALL + b * 4, call);
    }
    if (b == 0) {
        const u64 t0 = now();
        auto rf = __builtin_amdgcn_make_buffer_rsrc(myws + WS_FLAG, 0, NR * 128, 0x00020000);
        for (unsigned it = 1;; ++it) {
            const u32x4 v = __builtin_amdgcn_raw_buffer_load_b128(rf, lane * 16, 0, 17);
            const bool ok = (lane >> 3) == rank || (v.x >= call && v.y >= call && v.z >= call && v.w >= call);
            if (__all(ok)) break;
            if ((it & 255) == 0 && (ld_agent(myws + WS_TMO) == call || now() - t0 > DEADLINE_TICKS)) {
                st_agent(myws + WS_TMO, call);
                WAIT_VM(0);
                if (lane == 0) *(float*)(p.out[0] + (size_t)(rank * MLOC) * (NCOL * 4)) = __builtin_nanf("");
                break;
            }
        }
    }
}

extern "C" void kernel_launch(int rank, int nranks, ncclComm_t comm,
                              hipStream_t stream,
                              void* const* d_in, const int* in_sizes, int n_in,
                              void* d_out, int out_size,
                              const int* in_dtypes, int out_dtype,
                              void* const* d_out_all,
                              void* const* d_ws_all, size_t ws_bytes) {
    if (nranks != NR || (unsigned)rank >= (unsigned)NR || n_in < 4 || ws_bytes < WS_END ||
        in_sizes[0] != (int)X_BYTES || in_sizes[1] != (int)W_BYTES || out_size != 4096 * NCOL) {
        fprintf(stderr, "kernel_launch: unexpected problem shape\n");
        return;
    }
    HIP_CHECK(hipFuncSetAttribute((const void*)gemm_a2a, hipFuncAttributeMaxDynamicSharedMemorySize, LDS_BYTES));
    Params p;
    p.x = (const char*)d_in[0];
    p.w = (const char*)d_in[1];
    p.sx = (const float*)d_in[2];
    p.sw = (const float*)d_in[3];
    for (int i = 0; i < NR; ++i) { p.out[i] = (char*)d_out_all[(rank + i) % NR]; p.ws[i] = (char*)d_ws_all[(rank + i) % NR]; }
    p.dbg = nullptr;
    p.rank = rank;
    p.pad = 0;
    p.xs = (char*)d_ws_all[rank] + WS_XS;
    gemm_a2a<<<NWG, 768, LDS_BYTES, stream>>>(p);
    HIP_CHECK(hipGetLastError());
}
```
